```python
import numpy as np

kernel_name = "gpu_gemm_a2a_m4096_k4096_n16384_e4m3e4m3_dqrelu"


ABSMAX_RTOL = 0.03

COMPUTE_DTYPE = "float32"
EPILOGUE = "dequant_relu"
IN_QUANT = None
OUT_QUANT = None
IN_FP8 = "e4m3e4m3"
IN_DTYPES = {"x": "e4m3", "w_mat": "e4m3",
             "scale_x": "float32", "scale_w": "float32"}
WORLD_SIZE = 8

def _snap_fp8(x, mant_bits, max_abs, min_normal):
    a = np.abs(x.astype(np.float64))
    _m, _e = np.frexp(a)
    q = np.where(a >= min_normal, np.ldexp(1.0, _e - 1 - mant_bits),
                 min_normal * 2.0 ** -mant_bits)
    snapped = np.minimum(np.round(a / q) * q, max_abs)
    return (np.sign(x) * snapped).astype(np.float32)

def setup_inputs():

    x = _snap_fp8(np.random.randn(4096, 4096).astype(np.float32), 3, 448.0, 2.0 ** -6)
    w_mat = _snap_fp8(np.random.randn(4096, 16384).astype(np.float32), 3, 448.0, 2.0 ** -6)
    scale_x = (0.02 * (1.0 + np.random.rand(1))).astype(np.float32)
    scale_w = (0.02 * (1.0 + np.random.rand(1))).astype(np.float32)
    return {"x": x, "w_mat": w_mat, "scale_x": scale_x, "scale_w": scale_w}

def reference(x, w_mat, scale_x, scale_w):
    acc = (x @ w_mat).astype(np.float32)
    y = (acc * (scale_x * scale_w)).astype(np.float32)
    return np.maximum(y, 0.0).astype(np.float32)
```

```cpp
#include <hip/hip_runtime.h>
#include <rccl/rccl.h>
#include <cstdio>
#include <cstdint>

#define HIP_CHECK(expr) do { hipError_t _e = (expr); if (_e != hipSuccess) { \
    fprintf(stderr, "HIP error %s at %s:%d\n", hipGetErrorString(_e), __FILE__, __LINE__); \
    return; } } while (0)

namespace {

typedef int   v8i  __attribute__((ext_vector_type(8)));
typedef int   v4i  __attribute__((ext_vector_type(4)));
typedef int   v2i  __attribute__((ext_vector_type(2)));
typedef float v16f __attribute__((ext_vector_type(16)));

constexpr int NR    = 8;
constexpr int KDIM  = 4096;
constexpr int NFULL = 16384;
constexpr int NS    = 2048;
constexpr int BM = 256, BN = 128, BK = 128;
constexpr int NKS   = KDIM / BK;
#define CHST 2
constexpr int CH_ST = CHST;
constexpr int NCH   = NKS / CH_ST;
constexpr int A_BYTES = BM * BK;
constexpr int B_BYTES = BK * BN;
constexpr int ST_BYTES = A_BYTES + B_BYTES;
constexpr int NBUF = 3;
constexpr int SYNC_OFF = NBUF * ST_BYTES;
constexpr int LDS_BYTES = SYNC_OFF + 64;

constexpr int WGS = 256, THREADS = 704;
#define ST_BITS " sc0 sc1"
#define LAG 0
#define W1

constexpr size_t OFF_CTR   = 0;
constexpr size_t OFF_TMO   = 512;
constexpr size_t OFF_FLAGS = 4096;
constexpr size_t OFF_STAGE = 1u << 20;
constexpr size_t SLOT_BYTES = (size_t)NR * 2 * NKS * A_BYTES;
constexpr size_t WS_NEED   = OFF_STAGE + 2 * SLOT_BYTES;

constexpr unsigned long long DEADLINE_TICKS = 150000000ull;

template <int N> __device__ __forceinline__ void wait_vm() {
    asm volatile("s_waitcnt vmcnt(%0)" :: "n"(N) : "memory");
}
#define DS_B128(dst, addr, off) asm volatile("ds_read_b128 %0, %1 offset:" #off : "=v"(dst) : "v"(addr) : "memory")
#define DS_TR8(dst, addr, off)  asm volatile("ds_read_b64_tr_b8 %0, %1 offset:" #off : "=v"(dst) : "v"(addr) : "memory")

__device__ __forceinline__ unsigned lds_ld(unsigned addr) {
    unsigned v;
    asm volatile("ds_read_b32 %0, %1\n\ts_waitcnt lgkmcnt(0)" : "=v"(v) : "v"(addr) : "memory");
    return (unsigned)__builtin_amdgcn_readfirstlane(v);
}

struct Params {
    const unsigned char* x;
    const unsigned char* w;
    const float* sx;
    const float* sw;
    float* out;
    void* const* ws_all;
    unsigned char* ws;
    int rank;
};

template <int STEP> struct CommStep {
    static __device__ __forceinline__ void run(const v4i (&d)[NKS], unsigned char* dst, unsigned off,
                                               unsigned* flag, unsigned call) {
        if constexpr (STEP == 4) __builtin_amdgcn_s_waitcnt(0x0F70);
        unsigned o0 = off + (unsigned)STEP * A_BYTES;
        asm volatile("global_store_dwordx4 %0, %1, %2 sc0 sc1\n\ts_nop 1" :: "v"(o0), "v"(d[STEP]), "s"(dst) : "memory");
        wait_vm<0>();
        if constexpr ((STEP % CH_ST) == CH_ST - 1) {
            unsigned* f = flag + (STEP / CH_ST) * 32;
            asm volatile("global_store_dword %0, %1, off sc0 sc1\n\ts_nop 1" :: "v"(f), "v"(call) : "memory");
        }
        if constexpr (STEP + 1 < NKS) CommStep<STEP + 1>::run(d, dst, off, flag, call);
    }
};

__global__ void __launch_bounds__(THREADS, 3) fused_kernel(Params p) {
    extern __shared__ __attribute__((aligned(16))) unsigned char lds[];
    const int tid  = threadIdx.x;
    const int lane = tid & 63;
    const int wave = __builtin_amdgcn_readfirstlane(tid >> 6);
    const int b = blockIdx.x;
    const unsigned lds_base = (unsigned)(size_t)(__attribute__((address_space(3))) unsigned char*)lds;
    const unsigned a_cnt = lds_base + SYNC_OFF, a_avail = lds_base + SYNC_OFF + 16;

    const int xcd = b & 7, j = b >> 3;
    unsigned* sync_u = (unsigned*)(lds + SYNC_OFF);
    if (tid == 0) {
        unsigned t = __hip_atomic_fetch_add((unsigned*)(p.ws + OFF_CTR), 1u, __ATOMIC_RELAXED, __HIP_MEMORY_SCOPE_AGENT);
        sync_u[0] = 0u; sync_u[4] = 0u;
        sync_u[8] = t / WGS + 1u;
    }
    if (wave == 8 || wave == 9) {
        const int rg64 = xcd * 8 + (j >> 3) * 2 + (wave & 1);
        const int half = rg64 >> 5, rg = rg64 & 31;
        const int rowt = rg * 8 + (lane >> 3);
        const int c = (lane & 7) ^ ((rowt >> 1) & 7);
        const unsigned char* src = p.x + (size_t)(half * 256 + rowt) * KDIM + c * 16;
        v4i d[NKS];
#pragma unroll
        for (int i = 0; i < NKS; ++i) d[i] = *(const v4i*)(src + i * BK);
        const unsigned dest = (unsigned)(p.rank + 1 + ((j + xcd) & 7)) & 7u;
        unsigned long long wsv = (unsigned long long)p.ws_all[dest];
        wsv = ((unsigned long long)(unsigned)__builtin_amdgcn_readfirstlane((unsigned)(wsv >> 32)) << 32) |
              (unsigned long long)(unsigned)__builtin_amdgcn_readfirstlane((unsigned)wsv);
        unsigned char* wsd = (unsigned char*)wsv;
        __builtin_amdgcn_s_barrier();
        const unsigned call = lds_ld(lds_base + SYNC_OFF + 32);
        unsigned char* dst = wsd + OFF_STAGE + (call & 1u) * SLOT_BYTES + (size_t)(p.rank * 2 + half) * NKS * A_BYTES;
        unsigned* flag = (unsigned*)(wsd + OFF_FLAGS) + (size_t)(p.rank * 2 + half) * NCH * 32 + rg;
        unsigned off = (unsigned)(rg * 1024 + lane * 16);
        CommStep<0>::run(d, dst, off, flag, call);
        return;
    }
    __syncthreads();
    const unsigned call = __builtin_amdgcn_readfirstlane(sync_u[8]);
    const unsigned slot = call & 1u;
#define STAMP(i)
    STAMP(0);

    const int mt = 4 * (xcd & 3) + (j & 3);
    const int nt = 8 * (xcd >> 2) + (j >> 2);
    unsigned* tmo = (unsigned*)(p.ws + OFF_TMO);

    if (wave >= 8) {
        {
            const unsigned* f = (const unsigned*)(p.ws + OFF_FLAGS) + (size_t)mt * NCH * 32 + (lane & 31);
            const unsigned long long t0 = __builtin_amdgcn_s_memrealtime();
            bool dead = false;
            for (int ch = 0; ch < NCH; ++ch) {
                while (!dead) {
                    unsigned fv = __hip_atomic_load(f + ch * 32, __ATOMIC_RELAXED, __HIP_MEMORY_SCOPE_SYSTEM);
                    if (__builtin_amdgcn_ballot_w64(fv < call) == 0ull) break;
                    if (__hip_atomic_load(tmo, __ATOMIC_RELAXED, __HIP_MEMORY_SCOPE_AGENT) == call ||
                        __builtin_amdgcn_s_memrealtime() - t0 > DEADLINE_TICKS) {
                        __hip_atomic_store(tmo, call, __ATOMIC_RELAXED, __HIP_MEMORY_SCOPE_AGENT);
                        asm volatile("s_waitcnt vmcnt(0)" ::: "memory");
                        dead = true;
                    }
                    __builtin_amdgcn_s_sleep(1);
                }
                unsigned v = (unsigned)ch + 1u;
                asm volatile("ds_write_b32 %0, %1" :: "v"(a_avail), "v"(v) : "memory");
            }
        }
        return;
    }

    const unsigned char* a_src = p.ws + OFF_STAGE + slot * SLOT_BYTES + (size_t)mt * NKS * A_BYTES;
    const unsigned char* b_src = p.w + (size_t)p.rank * NS + nt * BN;

    const unsigned char* g_ptr;
    int g_step, g_piece, l_off;
    if (wave < 4) {
        g_ptr = a_src + wave * 8192 + lane * 16;
        g_step = A_BYTES; g_piece = 1024; l_off = wave * 8192;
    } else {
        const int wb = (wave - 4) & 1;
        const int kr = wb * 64 + (lane >> 3);
        const int ch = (lane & 7) ^ (((kr >> 1) & 3) << 1);
        g_ptr = b_src + (size_t)kr * NFULL + ch * 16;
        g_step = BK * NFULL; g_piece = 8 * NFULL; l_off = A_BYTES + wb * 8192;
    }
    auto issue_stage = [&](int s, int buf) {
        const unsigned char* g = g_ptr + (size_t)s * g_step;
        unsigned char* l = lds + buf * ST_BYTES + l_off;
        if (wave < 4) {
#pragma unroll
            for (int i = 0; i < 8; ++i)
                __builtin_amdgcn_global_load_lds((const unsigned*)(g + i * 1024), (unsigned*)(l + i * 1024), 16, 0, 17);
        } else if (wave < 6) {
#pragma unroll
            for (int i = 0; i < 8; ++i)
                __builtin_amdgcn_global_load_lds((const unsigned*)(g + (size_t)i * g_piece), (unsigned*)(l + i * 1024), 16, 0, 0);
        }
    };
    auto wait_avail = [&](int s) {
        const unsigned need = (unsigned)(s / CH_ST) + 1u;
        while (lds_ld(a_avail) < need) __builtin_amdgcn_s_sleep(1);
    };
    auto arrive = [&]() {
        if (lane == 0) {
            unsigned one = 1u;
            asm volatile("ds_add_u32 %0, %1" :: "v"(a_cnt), "v"(one) : "memory");
        }
    };

    const int wm = wave >> 1, wn = wave & 1;
    const int h = lane >> 5, r31 = lane & 31;
    const int ax = (r31 >> 1) & 7;
    const int a_base = (wm * 64 + r31) * 128;
    int a_off[2][2];
#pragma unroll
    for (int s = 0; s < 2; ++s)
#pragma unroll
        for (int e = 0; e < 2; ++e) a_off[s][e] = a_base + (((4 * s + 2 * h + e) ^ ax) << 4);
    const int g = lane >> 4, q = (lane & 15) >> 1, pp = lane & 1;
    int b_off[2];
#pragma unroll
    for (int nb = 0; nb < 2; ++nb)
        b_off[nb] = A_BYTES + (32 * h + q) * 128 + (((wn * 4 + nb * 2 + (g & 1)) ^ (((q >> 1) & 3) << 1)) << 4) + pp * 8;

    v16f acc[2][2];
#pragma unroll
    for (int i = 0; i < 2; ++i)
#pragma unroll
        for (int k = 0; k < 2; ++k)
#pragma unroll
            for (int r = 0; r < 16; ++r) acc[i][k][r] = 0.f;

    if (wave < 4) wait_avail(0);
    STAMP(1);
    issue_stage(0, 0);
    issue_stage(1, 1);
    wait_vm<8>();
    arrive();

    int buf = 0;
    for (int s = 0; s < NKS; ++s) {
        const int s2 = s + 2;
        int nbuf = buf + 2; if (nbuf >= NBUF) nbuf -= NBUF;
        {
            const unsigned target = 8u * (unsigned)(s + 1);
            while (lds_ld(a_cnt) < target) {}
        }
        bool issued = true;
        if (s2 < NKS) {
            if (wave < 4) issued = lds_ld(a_avail) >= (unsigned)(s2 / CH_ST) + 1u;
            if (issued) issue_stage(s2, nbuf);
        }

        const unsigned lb = lds_base + buf * ST_BYTES;
        v4i alo[2][2], ahi[2][2];
        v2i bt[2][2][4];
        {
            const unsigned a0 = lb + a_off[0][0], a1 = lb + a_off[0][1];
            const unsigned a2 = lb + a_off[1][0], a3 = lb + a_off[1][1];
            const unsigned b0 = lb + b_off[0], b1 = lb + b_off[1];
            DS_B128(alo[0][0], a0, 0);    DS_B128(ahi[0][0], a1, 0);
            DS_TR8(bt[0][0][0], b0, 0);   DS_TR8(bt[0][0][1], b0, 1024);
            DS_TR8(bt[0][0][2], b0, 2048); DS_TR8(bt[0][0][3], b0, 3072);
            DS_B128(alo[0][1], a0, 4096); DS_B128(ahi[0][1], a1, 4096);
            DS_TR8(bt[0][1][0], b1, 0);   DS_TR8(bt[0][1][1], b1, 1024);
            DS_TR8(bt[0][1][2], b1, 2048); DS_TR8(bt[0][1][3], b1, 3072);
            DS_B128(alo[1][0], a2, 0);    DS_B128(ahi[1][0], a3, 0);
            DS_TR8(bt[1][0][0], b0, 8192); DS_TR8(bt[1][0][1], b0, 9216);
            DS_TR8(bt[1][0][2], b0, 10240); DS_TR8(bt[1][0][3], b0, 11264);
            DS_B128(alo[1][1], a2, 4096); DS_B128(ahi[1][1], a3, 4096);
            DS_TR8(bt[1][1][0], b1, 8192); DS_TR8(bt[1][1][1], b1, 9216);
            DS_TR8(bt[1][1][2], b1, 10240); DS_TR8(bt[1][1][3], b1, 11264);
        }
#pragma unroll
        for (int ks = 0; ks < 2; ++ks) {
            if (ks == 0) {
                asm volatile("s_waitcnt lgkmcnt(12)" ::: "memory");
            } else {
                asm volatile("s_waitcnt lgkmcnt(0)" ::: "memory");
                if (s + 1 < NKS) {
                    if (s2 < NKS && issued) wait_vm<8>(); else wait_vm<0>();
                    arrive();
                }
            }
            __builtin_amdgcn_sched_barrier(0);
            v8i af[2], bf[2];
#pragma unroll
            for (int mb = 0; mb < 2; ++mb)
                af[mb] = __builtin_shufflevector(alo[ks][mb], ahi[ks][mb], 0, 1, 2, 3, 4, 5, 6, 7);
#pragma unroll
            for (int nb = 0; nb < 2; ++nb) {
                v4i lo = __builtin_shufflevector(bt[ks][nb][0], bt[ks][nb][1], 0, 1, 2, 3);
                v4i hi = __builtin_shufflevector(bt[ks][nb][2], bt[ks][nb][3], 0, 1, 2, 3);
                bf[nb] = __builtin_shufflevector(lo, hi, 0, 1, 2, 3, 4, 5, 6, 7);
            }
#pragma unroll
            for (int mb = 0; mb < 2; ++mb)
#pragma unroll
                for (int nb = 0; nb < 2; ++nb)
                    acc[mb][nb] = __builtin_amdgcn_mfma_scale_f32_32x32x64_f8f6f4(
                        af[mb], bf[nb], acc[mb][nb], 0, 0, 0, 0x7f7f7f7f, 0, 0x7f7f7f7f);
            __builtin_amdgcn_sched_barrier(0);
        }
        if (!issued) { wait_avail(s2); issue_stage(s2, nbuf); }
        buf = buf + 1; if (buf >= NBUF) buf = 0;
    }

    STAMP(2);
    const float scale = p.sx[0] * p.sw[0];
    const bool bad = __hip_atomic_load(tmo, __ATOMIC_RELAXED, __HIP_MEMORY_SCOPE_AGENT) == call;
    const float nanv = __builtin_nanf("");
    float* o = p.out + (size_t)(mt * BM + wm * 64 + 4 * h) * NS + nt * BN + wn * 64 + r31;
#pragma unroll
    for (int mb = 0; mb < 2; ++mb)
#pragma unroll
        for (int nb = 0; nb < 2; ++nb)
#pragma unroll
            for (int r = 0; r < 16; ++r) {
                const int row = mb * 32 + (r & 3) + 8 * (r >> 2);
                float v = fmaxf(acc[mb][nb][r] * scale, 0.f);
                o[(size_t)row * NS + nb * 32] = bad ? nanv : v;
            }
    STAMP(3);
}

}

extern "C" void kernel_launch(int rank, int nranks, ncclComm_t comm,
                              hipStream_t stream,
                              void* const* d_in, const int* in_sizes, int n_in,
                              void* d_out, int out_size,
                              const int* in_dtypes, int out_dtype,
                              void* const* d_out_all,
                              void* const* d_ws_all, size_t ws_bytes) {
    (void)comm; (void)in_sizes; (void)n_in; (void)out_size; (void)in_dtypes; (void)out_dtype; (void)d_out_all;
    if (nranks != NR || ws_bytes < WS_NEED || (unsigned)rank >= (unsigned)NR) {
        fprintf(stderr, "unexpected configuration\n");
        return;
    }
    HIP_CHECK(hipFuncSetAttribute((const void*)fused_kernel, hipFuncAttributeMaxDynamicSharedMemorySize, LDS_BYTES));
    Params p;
    p.x = (const unsigned char*)d_in[0];
    p.w = (const unsigned char*)d_in[1];
    p.sx = (const float*)d_in[2];
    p.sw = (const float*)d_in[3];
    p.out = (float*)d_out;
    p.ws_all = d_ws_all;
    p.ws = (unsigned char*)d_ws_all[rank];
    p.rank = rank;
    fused_kernel<<<WGS, THREADS, LDS_BYTES, stream>>>(p);
}
```

```cpp
#include <hip/hip_runtime.h>
#include <rccl/rccl.h>
#include <cstdio>
#include <cstdint>

#define HIP_CHECK(expr) do { hipError_t _e = (expr); if (_e != hipSuccess) { \
    fprintf(stderr, "HIP error %s at %s:%d\n", hipGetErrorString(_e), __FILE__, __LINE__); \
    return; } } while (0)

namespace {

typedef int   v8i  __attribute__((ext_vector_type(8)));
typedef int   v4i  __attribute__((ext_vector_type(4)));
typedef int   v2i  __attribute__((ext_vector_type(2)));
typedef float v16f __attribute__((ext_vector_type(16)));

constexpr int NR    = 8;
constexpr int KDIM  = 4096;
constexpr int NFULL = 16384;
constexpr int NS    = 2048;
constexpr int BM = 256, BN = 128, BK = 128;
constexpr int NKS   = KDIM / BK;
#define CHST 2
constexpr int CH_ST = CHST;
constexpr int NCH   = NKS / CH_ST;
constexpr unsigned Q_ITEMS = NCH * 32;
constexpr unsigned Q_WAVES = 32;
#define PACE 6
constexpr int A_BYTES = BM * BK;
constexpr int B_BYTES = BK * BN;
constexpr int ST_BYTES = A_BYTES + B_BYTES;
constexpr int NBUF = 3;
constexpr int SYNC_OFF = NBUF * ST_BYTES;
constexpr int LDS_BYTES = SYNC_OFF + 64;

constexpr int WGS = 256, THREADS = 704;
#define ST_BITS " sc0 sc1"
#define LAG 0
#define W1

constexpr size_t OFF_CTR   = 0;
constexpr size_t OFF_TMO   = 512;
constexpr size_t OFF_QUEUE = 4096;
constexpr size_t OFF_FLAGS = 8192;
constexpr size_t OFF_STAGE = 1u << 20;
constexpr size_t SLOT_BYTES = (size_t)NR * 2 * NKS * A_BYTES;
constexpr size_t WS_NEED   = OFF_STAGE + 2 * SLOT_BYTES;

constexpr unsigned long long DEADLINE_TICKS = 150000000ull;

template <int N> __device__ __forceinline__ void wait_vm() {
    asm volatile("s_waitcnt vmcnt(%0)" :: "n"(N) : "memory");
}
#define DS_B128(dst, addr, off) asm volatile("ds_read_b128 %0, %1 offset:" #off : "=v"(dst) : "v"(addr) : "memory")
#define DS_TR8(dst, addr, off)  asm volatile("ds_read_b64_tr_b8 %0, %1 offset:" #off : "=v"(dst) : "v"(addr) : "memory")

__device__ __forceinline__ unsigned lds_ld(unsigned addr) {
    unsigned v;
    asm volatile("ds_read_b32 %0, %1\n\ts_waitcnt lgkmcnt(0)" : "=v"(v) : "v"(addr) : "memory");
    return (unsigned)__builtin_amdgcn_readfirstlane(v);
}

struct Params {
    const unsigned char* x;
    const unsigned char* w;
    const float* sx;
    const float* sw;
    float* out;
    void* const* ws_all;
    unsigned char* ws;
    int rank;
};

__device__ __forceinline__ unsigned claim_issue(unsigned* q, unsigned inc) {
    unsigned r;
    const unsigned zero = 0u;
    asm volatile("s_mov_b64 exec, 1\n\tglobal_atomic_add %0, %1, %2, %3 sc0\n\ts_mov_b64 exec, -1"
                 : "=&v"(r) : "v"(zero), "v"(inc), "s"(q) : "memory");
    return r;
}
template <bool WAIT> __device__ __forceinline__ unsigned claim_get(unsigned r) {
    unsigned v;
    if constexpr (WAIT) asm volatile("s_waitcnt vmcnt(0)\n\tv_readfirstlane_b32 %0, %1\n\ts_nop 4" : "=s"(v) : "v"(r) : "memory");
    else                asm volatile("v_readfirstlane_b32 %0, %1\n\ts_nop 4" : "=s"(v) : "v"(r) : "memory");
    return v;
}
__device__ __forceinline__ void push_1k(unsigned off, v4i d, const unsigned char* base) {
    asm volatile("global_store_dwordx4 %0, %1, %2 sc0 sc1\n\ts_nop 1" :: "v"(off), "v"(d), "s"(base) : "memory");
}
__device__ __forceinline__ void push_flag(unsigned off, unsigned v, const unsigned char* base) {
    asm volatile("global_store_dword %0, %1, %2 sc0 sc1\n\ts_nop 1" :: "v"(off), "v"(v), "s"(base) : "memory");
}

__global__ void __launch_bounds__(THREADS, 3) fused_kernel(Params p) {
    extern __shared__ __attribute__((aligned(16))) unsigned char lds[];
    const int tid  = threadIdx.x;
    const int lane = tid & 63;
    const int wave = __builtin_amdgcn_readfirstlane(tid >> 6);
    const int b = blockIdx.x;
    const unsigned lds_base = (unsigned)(size_t)(__attribute__((address_space(3))) unsigned char*)lds;
    const unsigned a_cnt = lds_base + SYNC_OFF, a_avail = lds_base + SYNC_OFF + 16;

    const int xcd = b & 7, j = b >> 3;
    unsigned* ctr = (unsigned*)(p.ws + OFF_CTR);
    const unsigned a_exit = lds_base + SYNC_OFF + 32;
    if (tid == 0) {
        const unsigned z = 0u;
        asm volatile("ds_write_b32 %0, %1\n\tds_write_b32 %0, %1 offset:16\n\tds_write_b32 %0, %1 offset:32\n\t"
                     "s_waitcnt lgkmcnt(0)" :: "v"(a_cnt), "v"(z) : "memory");
    }
    auto wave_exit = [&]() {
        if (lane == 0) {
            unsigned old;
            const unsigned one = 1u;
            asm volatile("ds_add_rtn_u32 %0, %1, %2\n\ts_waitcnt lgkmcnt(0)" : "=v"(old) : "v"(a_exit), "v"(one) : "memory");
            if (old == (unsigned)(THREADS / 64 - 1))
                __hip_atomic_fetch_add(ctr, 1u, __ATOMIC_RELAXED, __HIP_MEMORY_SCOPE_AGENT);
        }
    };
    if (wave == 8 || wave == 9) {
        unsigned ctr_v = __hip_atomic_load(ctr, __ATOMIC_RELAXED, __HIP_MEMORY_SCOPE_AGENT);
        const int di = (j + xcd) & 7;
        if (di < NR - 1) {
            const int sq = wave & 1;
            unsigned* q = (unsigned*)(p.ws + OFF_QUEUE + (size_t)(di * 2 + sq) * 256);
            const unsigned long long wsv = (unsigned long long)p.ws_all[(unsigned)(p.rank + 1 + di) & 7u];
            const unsigned char* wsd = (const unsigned char*)(
                ((unsigned long long)(unsigned)__builtin_amdgcn_readfirstlane((unsigned)(wsv >> 32)) << 32) |
                (unsigned long long)(unsigned)__builtin_amdgcn_readfirstlane((unsigned)wsv));
            const int cch = (lane & 7) ^ ((sq * 4 + (lane >> 4)) & 7);
            const unsigned char* src = p.x + (size_t)(sq * 8 + (lane >> 3)) * KDIM + cch * 16;
            auto src_of = [&](unsigned it) { return src + (size_t)(it & 31u) * (16 * KDIM) + (it >> 5) * (CH_ST * BK); };
            unsigned cur = (unsigned)(xcd * 4 + (j >> 3));
            v4i d[CH_ST];
#pragma unroll
            for (int i = 0; i < CH_ST; ++i) d[i] = *(const v4i*)(src_of(cur) + i * BK);
            unsigned raw = claim_issue(q, 1u);
            __builtin_amdgcn_s_barrier();
            asm volatile("" : "+v"(ctr_v));
            const unsigned call = (unsigned)__builtin_amdgcn_readfirstlane(ctr_v) / WGS + 1u;
            const unsigned qoff = Q_WAVES - (call - 1u) * Q_ITEMS;
            const unsigned off0 = (unsigned)(OFF_STAGE + (size_t)p.rank * 2 * NKS * A_BYTES) + (call & 1u) * (unsigned)SLOT_BYTES +
                                  (unsigned)(sq * 1024 + lane * 16);
            const unsigned foff0 = (unsigned)OFF_FLAGS + (unsigned)(p.rank * 2 * NCH * 32 + sq) * 4u;
            unsigned nxt = claim_get<true>(raw) + qoff;
#pragma unroll
            for (int i = 0; i < CH_ST; ++i) asm volatile("" :: "v"(d[i]));
            unsigned pflag = 0u;
            while (true) {
                const bool more = nxt < Q_ITEMS;
                v4i e[CH_ST];
#pragma unroll
                for (int i = 0; i < CH_ST; ++i) e[i] = *(const v4i*)(src_of(more ? nxt : cur) + i * BK);
                raw = claim_issue(q, more ? 1u : 0u);
                const unsigned r = cur & 31u, ch = cur >> 5;
                const unsigned off = off0 + ((r >> 4) * NKS + ch * CH_ST) * (unsigned)A_BYTES + (r & 15u) * 2048u;
#pragma unroll
                for (int i = 0; i < CH_ST; ++i) {
                    push_1k(off + (unsigned)i * A_BYTES, d[i], wsd);
                    if (i == 0 && pflag != 0u) push_flag(pflag, call, wsd);
                    wait_vm<0>();
                    if (i + 1 < CH_ST) __builtin_amdgcn_s_sleep(PACE);
                }
                const unsigned nn = claim_get<false>(raw) + qoff;
                pflag = foff0 + ((r >> 4) * NCH * 32 + ch * 32 + (r & 15u) * 2u) * 4u;
                if (!more) break;
                __builtin_amdgcn_s_sleep(PACE);
                cur = nxt; nxt = nn;
#pragma unroll
                for (int i = 0; i < CH_ST; ++i) d[i] = e[i];
            }
            push_flag(pflag, call, wsd);
        } else {
            __builtin_amdgcn_s_barrier();
        }
        wave_exit();
        return;
    }
    __builtin_amdgcn_s_barrier();
    asm volatile("" ::: "memory");
    const unsigned ctr_v = __hip_atomic_load(ctr, __ATOMIC_RELAXED, __HIP_MEMORY_SCOPE_AGENT);
    const unsigned call = (unsigned)__builtin_amdgcn_readfirstlane(ctr_v) / WGS + 1u;
    const unsigned slot = call & 1u;
#define STAMP(i)
    STAMP(0);

    const int mt = 4 * (xcd & 3) + (j & 3);
    const int nt = 8 * (xcd >> 2) + (j >> 2);
    unsigned* tmo = (unsigned*)(p.ws + OFF_TMO);

    const bool own = (mt >> 1) == p.rank;
    if (wave >= 8) {
        if (own) {
            const unsigned v = NCH;
            asm volatile("ds_write_b32 %0, %1" :: "v"(a_avail), "v"(v) : "memory");
        } else {
            const unsigned* f = (const unsigned*)(p.ws + OFF_FLAGS) + (size_t)mt * NCH * 32 + (lane & 31);
            const unsigned long long t0 = __builtin_amdgcn_s_memrealtime();
            bool dead = false;
            for (int ch = 0; ch < NCH; ++ch) {
                for (unsigned it = 1; !dead; ++it) {
                    unsigned fv = __hip_atomic_load(f + ch * 32, __ATOMIC_RELAXED, __HIP_MEMORY_SCOPE_SYSTEM);
                    if (__builtin_amdgcn_ballot_w64(fv < call) == 0ull) break;
                    if ((it & 15u) == 0u &&
                        (__hip_atomic_load(tmo, __ATOMIC_RELAXED, __HIP_MEMORY_SCOPE_AGENT) == call ||
                         __builtin_amdgcn_s_memrealtime() - t0 > DEADLINE_TICKS)) {
                        __hip_atomic_store(tmo, call, __ATOMIC_RELAXED, __HIP_MEMORY_SCOPE_AGENT);
                        asm volatile("s_waitcnt vmcnt(0)" ::: "memory");
                        dead = true;
                    }
                }
                unsigned v = (unsigned)ch + 1u;
                asm volatile("ds_write_b32 %0, %1" :: "v"(a_avail), "v"(v) : "memory");
            }
        }
        wave_exit();
        return;
    }

    const unsigned char* a_src = p.ws + OFF_STAGE + slot * SLOT_BYTES + (size_t)mt * NKS * A_BYTES;
    const unsigned char* b_src = p.w + (size_t)p.rank * NS + nt * BN;

    const unsigned char* g_ptr;
    int g_odd, g_pair, g_step, l_off;
    if (wave < 4) {
        l_off = wave * 8192;
        if (own) {
            const int row = (mt & 1) * 256 + wave * 64 + (lane >> 3);
            const int ce = (lane & 7) ^ (lane >> 4), co = ce ^ 4;
            g_ptr = p.x + (size_t)row * KDIM + ce * 16;
            g_odd = 8 * KDIM + (co - ce) * 16; g_pair = 16 * KDIM; g_step = BK;
        } else {
            g_ptr = a_src + wave * 8192 + lane * 16;
            g_odd = 1024; g_pair = 2048; g_step = A_BYTES;
        }
    } else {
        const int wb = (wave - 4) & 1;
        const int kr = wb * 64 + (lane >> 3);
        const int ch = (lane & 7) ^ (((kr >> 1) & 3) << 1);
        g_ptr = b_src + (size_t)kr * NFULL + ch * 16;
        g_odd = 8 * NFULL; g_pair = 16 * NFULL; g_step = BK * NFULL; l_off = A_BYTES + wb * 8192;
    }
    auto issue_stage = [&](int s, int buf) {
        const unsigned char* g = g_ptr + (size_t)s * g_step;
        unsigned char* l = lds + buf * ST_BYTES + l_off;
        if (wave < 4) {
#pragma unroll
            for (int i = 0; i < 8; ++i)
                __builtin_amdgcn_global_load_lds((const unsigned*)(g + (i >> 1) * g_pair + (i & 1) * g_odd),
                                                 (unsigned*)(l + i * 1024), 16, 0, 17);
        } else if (wave < 6) {
#pragma unroll
            for (int i = 0; i < 8; ++i)
                __builtin_amdgcn_global_load_lds((const unsigned*)(g + (i >> 1) * g_pair + (i & 1) * g_odd),
                                                 (unsigned*)(l + i * 1024), 16, 0, 0);
        }
    };
    auto wait_avail = [&](int s) {
        const unsigned need = (unsigned)(s / CH_ST) + 1u;
        while (lds_ld(a_avail) < need) __builtin_amdgcn_s_sleep(1);
    };
    auto arrive = [&]() {
        if (lane == 0) {
            unsigned one = 1u;
            asm volatile("ds_add_u32 %0, %1" :: "v"(a_cnt), "v"(one) : "memory");
        }
    };

    const int wm = wave >> 1, wn = wave & 1;
    const int h = lane >> 5, r31 = lane & 31;
    const int ax = (r31 >> 1) & 7;
    const int a_base = (wm * 64 + r31) * 128;
    int a_off[2][2];
#pragma unroll
    for (int s = 0; s < 2; ++s)
#pragma unroll
        for (int e = 0; e < 2; ++e) a_off[s][e] = a_base + (((4 * s + 2 * h + e) ^ ax) << 4);
    const int g = lane >> 4, q = (lane & 15) >> 1, pp = lane & 1;
    int b_off[2];
#pragma unroll
    for (int nb = 0; nb < 2; ++nb)
        b_off[nb] = A_BYTES + (32 * h + q) * 128 + (((wn * 4 + nb * 2 + (g & 1)) ^ (((q >> 1) & 3) << 1)) << 4) + pp * 8;

    v16f acc[2][2];
#pragma unroll
    for (int i = 0; i < 2; ++i)
#pragma unroll
        for (int k = 0; k < 2; ++k)
#pragma unroll
            for (int r = 0; r < 16; ++r) acc[i][k][r] = 0.f;

    if (wave < 4) wait_avail(0);
    STAMP(1);
    issue_stage(0, 0);
    if (wave < 4) wait_avail(1);
    issue_stage(1, 1);
    wait_vm<8>();
    arrive();

    int buf = 0;
    for (int s = 0; s < NKS; ++s) {
        const int s2 = s + 2;
        int nbuf = buf + 2; if (nbuf >= NBUF) nbuf -= NBUF;
        {
            const unsigned target = 8u * (unsigned)(s + 1);
            while (lds_ld(a_cnt) < target) {}
        }
        bool issued = true;
        if (s2 < NKS) {
            if (wave < 4) issued = lds_ld(a_avail) >= (unsigned)(s2 / CH_ST) + 1u;
            if (issued) issue_stage(s2, nbuf);
        }

        const unsigned lb = lds_base + buf * ST_BYTES;
        v4i alo[2][2], ahi[2][2];
        v2i bt[2][2][4];
        {
            const unsigned a0 = lb + a_off[0][0], a1 = lb + a_off[0][1];
            const unsigned a2 = lb + a_off[1][0], a3 = lb + a_off[1][1];
            const unsigned b0 = lb + b_off[0], b1 = lb + b_off[1];
            DS_B128(alo[0][0], a0, 0);    DS_B128(ahi[0][0], a1, 0);
            DS_TR8(bt[0][0][0], b0, 0);   DS_TR8(bt[0][0][1], b0, 1024);
            DS_TR8(bt[0][0][2], b0, 2048); DS_TR8(bt[0][0][3], b0, 3072);
            DS_B128(alo[0][1], a0, 4096); DS_B128(ahi[0][1], a1, 4096);
            DS_TR8(bt[0][1][0], b1, 0);   DS_TR8(bt[0][1][1], b1, 1024);
            DS_TR8(bt[0][1][2], b1, 2048); DS_TR8(bt[0][1][3], b1, 3072);
            DS_B128(alo[1][0], a2, 0);    DS_B128(ahi[1][0], a3, 0);
            DS_TR8(bt[1][0][0], b0, 8192); DS_TR8(bt[1][0][1], b0, 9216);
            DS_TR8(bt[1][0][2], b0, 10240); DS_TR8(bt[1][0][3], b0, 11264);
            DS_B128(alo[1][1], a2, 4096); DS_B128(ahi[1][1], a3, 4096);
            DS_TR8(bt[1][1][0], b1, 8192); DS_TR8(bt[1][1][1], b1, 9216);
            DS_TR8(bt[1][1][2], b1, 10240); DS_TR8(bt[1][1][3], b1, 11264);
        }
#pragma unroll
        for (int ks = 0; ks < 2; ++ks) {
            if (ks == 0) {
                asm volatile("s_waitcnt lgkmcnt(12)" ::: "memory");
            } else {
                asm volatile("s_waitcnt lgkmcnt(0)" ::: "memory");
                if (s + 1 < NKS) {
                    if (s2 < NKS && issued) wait_vm<8>(); else wait_vm<0>();
                    arrive();
                }
            }
            __builtin_amdgcn_sched_barrier(0);
            v8i af[2], bf[2];
#pragma unroll
            for (int mb = 0; mb < 2; ++mb)
                af[mb] = __builtin_shufflevector(alo[ks][mb], ahi[ks][mb], 0, 1, 2, 3, 4, 5, 6, 7);
#pragma unroll
            for (int nb = 0; nb < 2; ++nb) {
                v4i lo = __builtin_shufflevector(bt[ks][nb][0], bt[ks][nb][1], 0, 1, 2, 3);
                v4i hi = __builtin_shufflevector(bt[ks][nb][2], bt[ks][nb][3], 0, 1, 2, 3);
                bf[nb] = __builtin_shufflevector(lo, hi, 0, 1, 2, 3, 4, 5, 6, 7);
            }
#pragma unroll
            for (int mb = 0; mb < 2; ++mb)
#pragma unroll
                for (int nb = 0; nb < 2; ++nb)
                    acc[mb][nb] = __builtin_amdgcn_mfma_scale_f32_32x32x64_f8f6f4(
                        af[mb], bf[nb], acc[mb][nb], 0, 0, 0, 0x7f7f7f7f, 0, 0x7f7f7f7f);
            __builtin_amdgcn_sched_barrier(0);
        }
        if (!issued) { wait_avail(s2); issue_stage(s2, nbuf); }
        buf = buf + 1; if (buf >= NBUF) buf = 0;
    }

    STAMP(2);
    const float scale = p.sx[0] * p.sw[0];
    const bool bad = __hip_atomic_load(tmo, __ATOMIC_RELAXED, __HIP_MEMORY_SCOPE_AGENT) == call;
    const float nanv = __builtin_nanf("");
    float* o = p.out + (size_t)(mt * BM + wm * 64 + 4 * h) * NS + nt * BN + wn * 64 + r31;
#pragma unroll
    for (int mb = 0; mb < 2; ++mb)
#pragma unroll
        for (int nb = 0; nb < 2; ++nb)
#pragma unroll
            for (int r = 0; r < 16; ++r) {
                const int row = mb * 32 + (r & 3) + 8 * (r >> 2);
                float v = fmaxf(acc[mb][nb][r] * scale, 0.f);
                __hip_atomic_store(&o[(size_t)row * NS + nb * 32], bad ? nanv : v, __ATOMIC_RELAXED, __HIP_MEMORY_SCOPE_AGENT);
            }
    STAMP(3);
    wave_exit();
}

}

extern "C" void kernel_launch(int rank, int nranks, ncclComm_t comm,
                              hipStream_t stream,
                              void* const* d_in, const int* in_sizes, int n_in,
                              void* d_out, int out_size,
                              const int* in_dtypes, int out_dtype,
                              void* const* d_out_all,
                              void* const* d_ws_all, size_t ws_bytes) {
    (void)comm; (void)in_sizes; (void)n_in; (void)out_size; (void)in_dtypes; (void)out_dtype; (void)d_out_all;
    if (nranks != NR || ws_bytes < WS_NEED || (unsigned)rank >= (unsigned)NR) {
        fprintf(stderr, "unexpected configuration\n");
        return;
    }
    HIP_CHECK(hipFuncSetAttribute((const void*)fused_kernel, hipFuncAttributeMaxDynamicSharedMemorySize, LDS_BYTES));
    Params p;
    p.x = (const unsigned char*)d_in[0];
    p.w = (const unsigned char*)d_in[1];
    p.sx = (const float*)d_in[2];
    p.sw = (const float*)d_in[3];
    p.out = (float*)d_out;
    p.ws_all = d_ws_all;
    p.ws = (unsigned char*)d_ws_all[rank];
    p.rank = rank;
    fused_kernel<<<WGS, THREADS, LDS_BYTES, stream>>>(p);
}
```

```cpp
#include <hip/hip_runtime.h>
#include <rccl/rccl.h>
#include <cstdio>
#include <cstdint>

#define HIP_CHECK(expr) do { hipError_t _e = (expr); if (_e != hipSuccess) { \
    fprintf(stderr, "HIP error %s at %s:%d\n", hipGetErrorString(_e), __FILE__, __LINE__); \
    return; } } while (0)

namespace {

typedef int   v8i  __attribute__((ext_vector_type(8)));
typedef int   v4i  __attribute__((ext_vector_type(4)));
typedef int   v2i  __attribute__((ext_vector_type(2)));
typedef float v16f __attribute__((ext_vector_type(16)));

constexpr int NR    = 8;
constexpr int KDIM  = 4096;
constexpr int NFULL = 16384;
constexpr int NS    = 2048;
constexpr int BM = 256, BN = 128, BK = 128;
constexpr int NKS   = KDIM / BK;
#define CHST 2
constexpr int CH_ST = CHST;
constexpr int NCH   = NKS / CH_ST;
constexpr unsigned Q_ITEMS = NCH * 32;
constexpr unsigned Q_WAVES = 32;
#define PACE 6
constexpr int A_BYTES = BM * BK;
constexpr int B_BYTES = BK * BN;
constexpr int ST_BYTES = A_BYTES + B_BYTES;
constexpr int NBUF = 3;
constexpr int SYNC_OFF = NBUF * ST_BYTES;
constexpr int LDS_BYTES = SYNC_OFF + 64;

constexpr int WGS = 256, THREADS = 704;
#define ST_BITS " sc0 sc1"
#define LAG 0
#define W1

constexpr size_t OFF_CTR   = 0;
constexpr size_t OFF_TMO   = 512;
constexpr size_t OFF_QUEUE = 4096;
constexpr size_t OFF_FLAGS = 8192;
constexpr size_t OFF_STAGE = 1u << 20;
constexpr size_t SLOT_BYTES = (size_t)NR * 2 * NKS * A_BYTES;
constexpr size_t WS_NEED   = OFF_STAGE + 2 * SLOT_BYTES;

constexpr unsigned long long DEADLINE_TICKS = 150000000ull;

template <int N> __device__ __forceinline__ void wait_vm() {
    asm volatile("s_waitcnt vmcnt(%0)" :: "n"(N) : "memory");
}
#define DS_B128(dst, addr, off) asm volatile("ds_read_b128 %0, %1 offset:" #off : "=v"(dst) : "v"(addr) : "memory")
#define DS_TR8(dst, addr, off)  asm volatile("ds_read_b64_tr_b8 %0, %1 offset:" #off : "=v"(dst) : "v"(addr) : "memory")

__device__ __forceinline__ unsigned lds_ld(unsigned addr) {
    unsigned v;
    asm volatile("ds_read_b32 %0, %1\n\ts_waitcnt lgkmcnt(0)" : "=v"(v) : "v"(addr) : "memory");
    return (unsigned)__builtin_amdgcn_readfirstlane(v);
}

struct Params {
    const unsigned char* x;
    const unsigned char* w;
    const float* sx;
    const float* sw;
    float* out;
    void* const* ws_all;
    unsigned char* ws;
    int rank;
};

__device__ __forceinline__ unsigned claim_issue(unsigned* q, unsigned inc) {
    unsigned r;
    const unsigned zero = 0u;
    asm volatile("s_mov_b64 exec, 1\n\tglobal_atomic_add %0, %1, %2, %3 sc0\n\ts_mov_b64 exec, -1"
                 : "=&v"(r) : "v"(zero), "v"(inc), "s"(q) : "memory");
    return r;
}
template <bool WAIT> __device__ __forceinline__ unsigned claim_get(unsigned r) {
    unsigned v;
    if constexpr (WAIT) asm volatile("s_waitcnt vmcnt(0)\n\tv_readfirstlane_b32 %0, %1\n\ts_nop 4" : "=s"(v) : "v"(r) : "memory");
    else                asm volatile("v_readfirstlane_b32 %0, %1\n\ts_nop 4" : "=s"(v) : "v"(r) : "memory");
    return v;
}
__device__ __forceinline__ void push_1k(unsigned off, v4i d, const unsigned char* base) {
    asm volatile("global_store_dwordx4 %0, %1, %2 sc0 sc1\n\ts_nop 1" :: "v"(off), "v"(d), "s"(base) : "memory");
}
__device__ __forceinline__ void push_flag(unsigned off, unsigned v, const unsigned char* base) {
    asm volatile("global_store_dword %0, %1, %2 sc0 sc1\n\ts_nop 1" :: "v"(off), "v"(v), "s"(base) : "memory");
}

__global__ void __launch_bounds__(THREADS, 3) fused_kernel(Params p) {
    extern __shared__ __attribute__((aligned(16))) unsigned char lds[];
    const int tid  = threadIdx.x;
    const int lane = tid & 63;
    const int wave = __builtin_amdgcn_readfirstlane(tid >> 6);
    const int b = blockIdx.x;
    const unsigned lds_base = (unsigned)(size_t)(__attribute__((address_space(3))) unsigned char*)lds;
    const unsigned a_cnt = lds_base + SYNC_OFF, a_avail = lds_base + SYNC_OFF + 16;

    const int xcd = b & 7, j = b >> 3;
    unsigned* ctr = (unsigned*)(p.ws + OFF_CTR);
    const unsigned a_exit = lds_base + SYNC_OFF + 32;
    if (tid == 0) {
        const unsigned z = 0u;
        asm volatile("ds_write_b32 %0, %1\n\tds_write_b32 %0, %1 offset:16\n\tds_write_b32 %0, %1 offset:32\n\t"
                     "s_waitcnt lgkmcnt(0)" :: "v"(a_cnt), "v"(z) : "memory");
    }
    auto wave_exit = [&]() {
        if (lane == 0) {
            unsigned old;
            const unsigned one = 1u;
            asm volatile("ds_add_rtn_u32 %0, %1, %2\n\ts_waitcnt lgkmcnt(0)" : "=v"(old) : "v"(a_exit), "v"(one) : "memory");
            if (old == (unsigned)(THREADS / 64 - 1))
                __hip_atomic_fetch_add(ctr, 1u, __ATOMIC_RELAXED, __HIP_MEMORY_SCOPE_AGENT);
        }
    };
    if (wave == 8 || wave == 9) {
        unsigned ctr_v = __hip_atomic_load(ctr, __ATOMIC_RELAXED, __HIP_MEMORY_SCOPE_AGENT);
        const int di = (j + xcd) & 7;
        if (di < NR - 1) {
            const int sq = wave & 1;
            unsigned* q = (unsigned*)(p.ws + OFF_QUEUE + (size_t)(di * 2 + sq) * 256);
            const unsigned long long wsv = (unsigned long long)p.ws_all[(unsigned)(p.rank + 1 + di) & 7u];
            const unsigned char* wsd = (const unsigned char*)(
                ((unsigned long long)(unsigned)__builtin_amdgcn_readfirstlane((unsigned)(wsv >> 32)) << 32) |
                (unsigned long long)(unsigned)__builtin_amdgcn_readfirstlane((unsigned)wsv));
            const int cch = (lane & 7) ^ ((sq * 4 + (lane >> 4)) & 7);
            const unsigned char* src = p.x + (size_t)(sq * 8 + (lane >> 3)) * KDIM + cch * 16;
            auto src_of = [&](unsigned it) { return src + (size_t)(it & 31u) * (16 * KDIM) + (it >> 5) * (CH_ST * BK); };
            unsigned cur = (unsigned)(xcd * 4 + (j >> 3));
            v4i d[CH_ST];
#pragma unroll
            for (int i = 0; i < CH_ST; ++i) d[i] = *(const v4i*)(src_of(cur) + i * BK);
            __builtin_amdgcn_s_barrier();
            asm volatile("" : "+v"(ctr_v));
            const unsigned call = (unsigned)__builtin_amdgcn_readfirstlane(ctr_v) / WGS + 1u;
            const unsigned qoff = Q_WAVES - (call - 1u) * Q_ITEMS;
            const unsigned off0 = (unsigned)(OFF_STAGE + (size_t)p.rank * 2 * NKS * A_BYTES) + (call & 1u) * (unsigned)SLOT_BYTES +
                                  (unsigned)(sq * 1024 + lane * 16);
            const unsigned foff0 = (unsigned)OFF_FLAGS + (unsigned)(p.rank * 2 * NCH * 32 + sq) * 4u;
            static_assert(CH_ST == 2, "sender loop is written for two k-stages per item");
            asm volatile("" :: "v"(d[0]), "v"(d[1]));
            unsigned raw = claim_issue(q, 1u);
            unsigned pflag = 0u;
            while (true) {
                const unsigned r = cur & 31u, ch = cur >> 5;
                const unsigned off = off0 + ((r >> 4) * NKS + ch * CH_ST) * (unsigned)A_BYTES + (r & 15u) * 2048u;
                push_1k(off, d[0], wsd);
                if (pflag != 0u) push_flag(pflag, call, wsd);
                const unsigned nxt = claim_get<true>(raw) + qoff;
                const bool more = nxt < Q_ITEMS;
                v4i e[CH_ST];
#pragma unroll
                for (int i = 0; i < CH_ST; ++i) e[i] = *(const v4i*)(src_of(more ? nxt : cur) + i * BK);
                raw = claim_issue(q, more ? 1u : 0u);
                __builtin_amdgcn_s_sleep(PACE);
                push_1k(off + (unsigned)A_BYTES, d[1], wsd);
                wait_vm<0>();
                pflag = foff0 + ((r >> 4) * NCH * 32 + ch * 32 + (r & 15u) * 2u) * 4u;
                if (!more) break;
                __builtin_amdgcn_s_sleep(PACE);
                cur = nxt;
#pragma unroll
                for (int i = 0; i < CH_ST; ++i) d[i] = e[i];
            }
            push_flag(pflag, call, wsd);
        } else {
            __builtin_amdgcn_s_barrier();
        }
        wave_exit();
        return;
    }
    __builtin_amdgcn_s_barrier();
    asm volatile("" ::: "memory");
    const unsigned ctr_v = __hip_atomic_load(ctr, __ATOMIC_RELAXED, __HIP_MEMORY_SCOPE_AGENT);
    const unsigned call = (unsigned)__builtin_amdgcn_readfirstlane(ctr_v) / WGS + 1u;
    const unsigned slot = call & 1u;
#define STAMP(i)
    STAMP(0);

    const int mt = 4 * (xcd & 3) + (j & 3);
    const int nt = 8 * (xcd >> 2) + (j >> 2);
    unsigned* tmo = (unsigned*)(p.ws + OFF_TMO);

    const bool own = (mt >> 1) == p.rank;
    if (wave >= 8) {
        if (own) {
            const unsigned v = NCH;
            asm volatile("ds_write_b32 %0, %1" :: "v"(a_avail), "v"(v) : "memory");
        } else {
            const unsigned* f = (const unsigned*)(p.ws + OFF_FLAGS) + (size_t)mt * NCH * 32 + (lane & 31);
            const unsigned long long t0 = __builtin_amdgcn_s_memrealtime();
            bool dead = false;
            for (int ch = 0; ch < NCH; ++ch) {
                for (unsigned it = 1; !dead; ++it) {
                    unsigned fv = __hip_atomic_load(f + ch * 32, __ATOMIC_RELAXED, __HIP_MEMORY_SCOPE_SYSTEM);
                    if (__builtin_amdgcn_ballot_w64(fv < call) == 0ull) break;
                    if ((it & 15u) == 0u &&
                        (__hip_atomic_load(tmo, __ATOMIC_RELAXED, __HIP_MEMORY_SCOPE_AGENT) == call ||
                         __builtin_amdgcn_s_memrealtime() - t0 > DEADLINE_TICKS)) {
                        __hip_atomic_store(tmo, call, __ATOMIC_RELAXED, __HIP_MEMORY_SCOPE_AGENT);
                        asm volatile("s_waitcnt vmcnt(0)" ::: "memory");
                        dead = true;
                    }
                }
                unsigned v = (unsigned)ch + 1u;
                asm volatile("ds_write_b32 %0, %1" :: "v"(a_avail), "v"(v) : "memory");
            }
        }
        wave_exit();
        return;
    }

    const unsigned char* a_src = p.ws + OFF_STAGE + slot * SLOT_BYTES + (size_t)mt * NKS * A_BYTES;
    const unsigned char* b_src = p.w + (size_t)p.rank * NS + nt * BN;

    const unsigned char* g_ptr;
    int g_odd, g_pair, g_step, l_off;
    if (wave < 4) {
        l_off = wave * 8192;
        if (own) {
            const int row = (mt & 1) * 256 + wave * 64 + (lane >> 3);
            const int ce = (lane & 7) ^ (lane >> 4), co = ce ^ 4;
            g_ptr = p.x + (size_t)row * KDIM + ce * 16;
            g_odd = 8 * KDIM + (co - ce) * 16; g_pair = 16 * KDIM; g_step = BK;
        } else {
            g_ptr = a_src + wave * 8192 + lane * 16;
            g_odd = 1024; g_pair = 2048; g_step = A_BYTES;
        }
    } else {
        const int wb = (wave - 4) & 1;
        const int kr = wb * 64 + (lane >> 3);
        const int ch = (lane & 7) ^ (((kr >> 1) & 3) << 1);
        g_ptr = b_src + (size_t)kr * NFULL + ch * 16;
        g_odd = 8 * NFULL; g_pair = 16 * NFULL; g_step = BK * NFULL; l_off = A_BYTES + wb * 8192;
    }
    auto issue_stage = [&](int s, int buf) {
        const unsigned char* g = g_ptr + (size_t)s * g_step;
        unsigned char* l = lds + buf * ST_BYTES + l_off;
        if (wave < 4) {
#pragma unroll
            for (int i = 0; i < 8; ++i)
                __builtin_amdgcn_global_load_lds((const unsigned*)(g + (i >> 1) * g_pair + (i & 1) * g_odd),
                                                 (unsigned*)(l + i * 1024), 16, 0, 17);
        } else if (wave < 6) {
#pragma unroll
            for (int i = 0; i < 8; ++i)
                __builtin_amdgcn_global_load_lds((const unsigned*)(g + (i >> 1) * g_pair + (i & 1) * g_odd),
                                                 (unsigned*)(l + i * 1024), 16, 0, 0);
        }
    };
    auto wait_avail = [&](int s) {
        const unsigned need = (unsigned)(s / CH_ST) + 1u;
        while (lds_ld(a_avail) < need) __builtin_amdgcn_s_sleep(1);
    };
    auto arrive = [&]() {
        if (lane == 0) {
            unsigned one = 1u;
            asm volatile("ds_add_u32 %0, %1" :: "v"(a_cnt), "v"(one) : "memory");
        }
    };

    const int wm = wave >> 1, wn = wave & 1;
    const int h = lane >> 5, r31 = lane & 31;
    const int ax = (r31 >> 1) & 7;
    const int a_base = (wm * 64 + r31) * 128;
    int a_off[2][2];
#pragma unroll
    for (int s = 0; s < 2; ++s)
#pragma unroll
        for (int e = 0; e < 2; ++e) a_off[s][e] = a_base + (((4 * s + 2 * h + e) ^ ax) << 4);
    const int g = lane >> 4, q = (lane & 15) >> 1, pp = lane & 1;
    int b_off[2];
#pragma unroll
    for (int nb = 0; nb < 2; ++nb)
        b_off[nb] = A_BYTES + (32 * h + q) * 128 + (((wn * 4 + nb * 2 + (g & 1)) ^ (((q >> 1) & 3) << 1)) << 4) + pp * 8;

    v16f acc[2][2];
#pragma unroll
    for (int i = 0; i < 2; ++i)
#pragma unroll
        for (int k = 0; k < 2; ++k)
#pragma unroll
            for (int r = 0; r < 16; ++r) acc[i][k][r] = 0.f;

    if (wave < 4) wait_avail(0);
    STAMP(1);
    issue_stage(0, 0);
    if (wave < 4) wait_avail(1);
    issue_stage(1, 1);
    wait_vm<8>();
    arrive();

    int buf = 0;
    for (int s = 0; s < NKS; ++s) {
        const int s2 = s + 2;
        int nbuf = buf + 2; if (nbuf >= NBUF) nbuf -= NBUF;
        {
            const unsigned target = 8u * (unsigned)(s + 1);
            while (lds_ld(a_cnt) < target) {}
        }
        bool issued = true;
        if (s2 < NKS) {
            if (wave < 4) issued = lds_ld(a_avail) >= (unsigned)(s2 / CH_ST) + 1u;
            if (issued) issue_stage(s2, nbuf);
        }

        const unsigned lb = lds_base + buf * ST_BYTES;
        v4i alo[2][2], ahi[2][2];
        v2i bt[2][2][4];
        {
            const unsigned a0 = lb + a_off[0][0], a1 = lb + a_off[0][1];
            const unsigned a2 = lb + a_off[1][0], a3 = lb + a_off[1][1];
            const unsigned b0 = lb + b_off[0], b1 = lb + b_off[1];
            DS_B128(alo[0][0], a0, 0);    DS_B128(ahi[0][0], a1, 0);
            DS_TR8(bt[0][0][0], b0, 0);   DS_TR8(bt[0][0][1], b0, 1024);
            DS_TR8(bt[0][0][2], b0, 2048); DS_TR8(bt[0][0][3], b0, 3072);
            DS_B128(alo[0][1], a0, 4096); DS_B128(ahi[0][1], a1, 4096);
            DS_TR8(bt[0][1][0], b1, 0);   DS_TR8(bt[0][1][1], b1, 1024);
            DS_TR8(bt[0][1][2], b1, 2048); DS_TR8(bt[0][1][3], b1, 3072);
            DS_B128(alo[1][0], a2, 0);    DS_B128(ahi[1][0], a3, 0);
            DS_TR8(bt[1][0][0], b0, 8192); DS_TR8(bt[1][0][1], b0, 9216);
            DS_TR8(bt[1][0][2], b0, 10240); DS_TR8(bt[1][0][3], b0, 11264);
            DS_B128(alo[1][1], a2, 4096); DS_B128(ahi[1][1], a3, 4096);
            DS_TR8(bt[1][1][0], b1, 8192); DS_TR8(bt[1][1][1], b1, 9216);
            DS_TR8(bt[1][1][2], b1, 10240); DS_TR8(bt[1][1][3], b1, 11264);
        }
#pragma unroll
        for (int ks = 0; ks < 2; ++ks) {
            if (ks == 0) {
                asm volatile("s_waitcnt lgkmcnt(12)" ::: "memory");
            } else {
                asm volatile("s_waitcnt lgkmcnt(0)" ::: "memory");
                if (s + 1 < NKS) {
                    if (s2 < NKS && issued) wait_vm<8>(); else wait_vm<0>();
                    arrive();
                }
            }
            __builtin_amdgcn_sched_barrier(0);
            v8i af[2], bf[2];
#pragma unroll
            for (int mb = 0; mb < 2; ++mb)
                af[mb] = __builtin_shufflevector(alo[ks][mb], ahi[ks][mb], 0, 1, 2, 3, 4, 5, 6, 7);
#pragma unroll
            for (int nb = 0; nb < 2; ++nb) {
                v4i lo = __builtin_shufflevector(bt[ks][nb][0], bt[ks][nb][1], 0, 1, 2, 3);
                v4i hi = __builtin_shufflevector(bt[ks][nb][2], bt[ks][nb][3], 0, 1, 2, 3);
                bf[nb] = __builtin_shufflevector(lo, hi, 0, 1, 2, 3, 4, 5, 6, 7);
            }
#pragma unroll
            for (int mb = 0; mb < 2; ++mb)
#pragma unroll
                for (int nb = 0; nb < 2; ++nb)
                    acc[mb][nb] = __builtin_amdgcn_mfma_scale_f32_32x32x64_f8f6f4(
                        af[mb], bf[nb], acc[mb][nb], 0, 0, 0, 0x7f7f7f7f, 0, 0x7f7f7f7f);
            __builtin_amdgcn_sched_barrier(0);
        }
        if (!issued) { wait_avail(s2); issue_stage(s2, nbuf); }
        buf = buf + 1; if (buf >= NBUF) buf = 0;
    }

    STAMP(2);
    const float scale = p.sx[0] * p.sw[0];
    const bool bad = __hip_atomic_load(tmo, __ATOMIC_RELAXED, __HIP_MEMORY_SCOPE_AGENT) == call;
    const float nanv = __builtin_nanf("");
    float* o = p.out + (size_t)(mt * BM + wm * 64 + 4 * h) * NS + nt * BN + wn * 64 + r31;
#pragma unroll
    for (int mb = 0; mb < 2; ++mb)
#pragma unroll
        for (int nb = 0; nb < 2; ++nb)
#pragma unroll
            for (int r = 0; r < 16; ++r) {
                const int row = mb * 32 + (r & 3) + 8 * (r >> 2);
                float v = fmaxf(acc[mb][nb][r] * scale, 0.f);
                __hip_atomic_store(&o[(size_t)row * NS + nb * 32], bad ? nanv : v, __ATOMIC_RELAXED, __HIP_MEMORY_SCOPE_AGENT);
            }
    STAMP(3);
    wave_exit();
}

}

extern "C" void kernel_launch(int rank, int nranks, ncclComm_t comm,
                              hipStream_t stream,
                              void* const* d_in, const int* in_sizes, int n_in,
                              void* d_out, int out_size,
                              const int* in_dtypes, int out_dtype,
                              void* const* d_out_all,
                              void* const* d_ws_all, size_t ws_bytes) {
    (void)comm; (void)in_sizes; (void)n_in; (void)out_size; (void)in_dtypes; (void)out_dtype; (void)d_out_all;
    if (nranks != NR || ws_bytes < WS_NEED || (unsigned)rank >= (unsigned)NR) {
        fprintf(stderr, "unexpected configuration\n");
        return;
    }
    HIP_CHECK(hipFuncSetAttribute((const void*)fused_kernel, hipFuncAttributeMaxDynamicSharedMemorySize, LDS_BYTES));
    Params p;
    p.x = (const unsigned char*)d_in[0];
    p.w = (const unsigned char*)d_in[1];
    p.sx = (const float*)d_in[2];
    p.sw = (const float*)d_in[3];
    p.out = (float*)d_out;
    p.ws_all = d_ws_all;
    p.ws = (unsigned char*)d_ws_all[rank];
    p.rank = rank;
    fused_kernel<<<WGS, THREADS, LDS_BYTES, stream>>>(p);
}
```

```cpp
#include <hip/hip_runtime.h>
#include <rccl/rccl.h>
#include <cstdio>
#include <cstdint>

#define HIP_CHECK(expr) do { hipError_t _e = (expr); if (_e != hipSuccess) { \
    fprintf(stderr, "HIP error %s at %s:%d\n", hipGetErrorString(_e), __FILE__, __LINE__); \
    return; } } while (0)

namespace {

typedef int   v8i  __attribute__((ext_vector_type(8)));
typedef int   v4i  __attribute__((ext_vector_type(4)));
typedef int   v2i  __attribute__((ext_vector_type(2)));
typedef float v16f __attribute__((ext_vector_type(16)));

constexpr int NR    = 8;
constexpr int KDIM  = 4096;
constexpr int NFULL = 16384;
constexpr int NS    = 2048;
constexpr int BM = 256, BN = 128, BK = 128;
constexpr int NKS   = KDIM / BK;
#define CHST 2
constexpr int CH_ST = CHST;
constexpr int NCH   = NKS / CH_ST;
constexpr unsigned Q_ITEMS = NCH * 32;
constexpr unsigned Q_WAVES = 32;
#define PACE 6
#define AGG
#define POLL_GAP 16
#define POLL_SLEEP 8
#define POLL_HGAP POLL_SLEEP
constexpr int A_BYTES = BM * BK;
constexpr int B_BYTES = BK * BN;
constexpr int ST_BYTES = A_BYTES + B_BYTES;
constexpr int NBUF = 3;
constexpr int SYNC_OFF = NBUF * ST_BYTES;
constexpr int LDS_BYTES = SYNC_OFF + 64;

constexpr int WGS = 256, THREADS = 704;
#define ST_BITS " sc0 sc1"
#define LAG 0
#define W1

constexpr size_t OFF_CTR   = 0;
constexpr size_t OFF_TMO   = 512;
constexpr size_t OFF_QUEUE = 4096;
constexpr size_t OFF_FLAGS = 8192;
constexpr size_t OFF_SCNT  = 65536;
constexpr size_t OFF_STAGE = 1u << 20;
constexpr size_t SLOT_BYTES = (size_t)NR * 2 * NKS * A_BYTES;
constexpr size_t WS_NEED   = OFF_STAGE + 2 * SLOT_BYTES;

constexpr unsigned long long DEADLINE_TICKS = 150000000ull;

template <int N> __device__ __forceinline__ void wait_vm() {
    asm volatile("s_waitcnt vmcnt(%0)" :: "n"(N) : "memory");
}
#define DS_B128(dst, addr, off) asm volatile("ds_read_b128 %0, %1 offset:" #off : "=v"(dst) : "v"(addr) : "memory")
#define DS_TR8(dst, addr, off)  asm volatile("ds_read_b64_tr_b8 %0, %1 offset:" #off : "=v"(dst) : "v"(addr) : "memory")

__device__ __forceinline__ unsigned lds_ld(unsigned addr) {
    unsigned v;
    asm volatile("ds_read_b32 %0, %1\n\ts_waitcnt lgkmcnt(0)" : "=v"(v) : "v"(addr) : "memory");
    return (unsigned)__builtin_amdgcn_readfirstlane(v);
}

struct Params {
    const unsigned char* x;
    const unsigned char* w;
    const float* sx;
    const float* sw;
    float* out;
    void* const* ws_all;
    unsigned char* ws;
    int rank;
};

__device__ __forceinline__ unsigned claim_issue(unsigned* q, unsigned inc) {
    unsigned r;
    const unsigned zero = 0u;
    asm volatile("s_mov_b64 exec, 1\n\tglobal_atomic_add %0, %1, %2, %3 sc0\n\ts_mov_b64 exec, -1"
                 : "=&v"(r) : "v"(zero), "v"(inc), "s"(q) : "memory");
    return r;
}
template <bool WAIT> __device__ __forceinline__ unsigned claim_get(unsigned r) {
    unsigned v;
    if constexpr (WAIT) asm volatile("s_waitcnt vmcnt(0)\n\tv_readfirstlane_b32 %0, %1\n\ts_nop 4" : "=s"(v) : "v"(r) : "memory");
    else                asm volatile("v_readfirstlane_b32 %0, %1\n\ts_nop 4" : "=s"(v) : "v"(r) : "memory");
    return v;
}
__device__ __forceinline__ void push_1k(unsigned off, v4i d, const unsigned char* base) {
    asm volatile("global_store_dwordx4 %0, %1, %2 sc0 sc1\n\ts_nop 1" :: "v"(off), "v"(d), "s"(base) : "memory");
}
__device__ __forceinline__ void push_flag(unsigned off, unsigned v, const unsigned char* base) {
    asm volatile("global_store_dword %0, %1, %2 sc0 sc1\n\ts_nop 1" :: "v"(off), "v"(v), "s"(base) : "memory");
}

__global__ void __launch_bounds__(THREADS, 3) fused_kernel(Params p) {
    extern __shared__ __attribute__((aligned(16))) unsigned char lds[];
    const int tid  = threadIdx.x;
    const int lane = tid & 63;
    const int wave = __builtin_amdgcn_readfirstlane(tid >> 6);
    const int b = blockIdx.x;
    const unsigned lds_base = (unsigned)(size_t)(__attribute__((address_space(3))) unsigned char*)lds;
    const unsigned a_cnt = lds_base + SYNC_OFF, a_avail = lds_base + SYNC_OFF + 16;

    const int xcd = b & 7, j = b >> 3;
    unsigned* ctr = (unsigned*)(p.ws + OFF_CTR);
    const unsigned a_exit = lds_base + SYNC_OFF + 32;
    if (tid == 0) {
        const unsigned z = 0u;
        asm volatile("ds_write_b32 %0, %1\n\tds_write_b32 %0, %1 offset:16\n\tds_write_b32 %0, %1 offset:32\n\t"
                     "s_waitcnt lgkmcnt(0)" :: "v"(a_cnt), "v"(z) : "memory");
    }
    auto wave_exit = [&]() {
        if (lane == 0) {
            unsigned old;
            const unsigned one = 1u;
            asm volatile("ds_add_rtn_u32 %0, %1, %2\n\ts_waitcnt lgkmcnt(0)" : "=v"(old) : "v"(a_exit), "v"(one) : "memory");
            if (old == (unsigned)(THREADS / 64 - 1))
                __hip_atomic_fetch_add(ctr, 1u, __ATOMIC_RELAXED, __HIP_MEMORY_SCOPE_AGENT);
        }
    };
    if (wave == 8 || wave == 9) {
        unsigned ctr_v = __hip_atomic_load(ctr, __ATOMIC_RELAXED, __HIP_MEMORY_SCOPE_AGENT);
        const int di = (j + xcd) & 7;
        if (di < NR - 1) {
            const int sq = wave & 1;
            unsigned* q = (unsigned*)(p.ws + OFF_QUEUE + (size_t)(di * 2 + sq) * 256);
            const unsigned long long wsv = (unsigned long long)p.ws_all[(unsigned)(p.rank + 1 + di) & 7u];
            const unsigned char* wsd = (const unsigned char*)(
                ((unsigned long long)(unsigned)__builtin_amdgcn_readfirstlane((unsigned)(wsv >> 32)) << 32) |
                (unsigned long long)(unsigned)__builtin_amdgcn_readfirstlane((unsigned)wsv));
            const unsigned lsrc = (unsigned)((sq * 16 + (lane >> 3)) * KDIM + (((lane & 7) ^ (lane >> 4)) << 4));
            auto load_item = [&](unsigned it, v4i* dst) {
                const unsigned char* sp = p.x + (size_t)((it & 15u) * (32u * KDIM) + (it >> 4) * BK);
                dst[0] = *(const v4i*)(sp + lsrc);
                dst[1] = *(const v4i*)(sp + 8 * KDIM + (lsrc ^ 64u));
            };
            unsigned cur = (unsigned)(xcd * 4 + (j >> 3));
            v4i d[2];
            load_item(cur, d);
            __builtin_amdgcn_s_barrier();
            asm volatile("" : "+v"(ctr_v));
            const unsigned call = (unsigned)__builtin_amdgcn_readfirstlane(ctr_v) / WGS + 1u;
            const unsigned qoff = Q_WAVES - (call - 1u) * Q_ITEMS;
            const unsigned off0 = (unsigned)(OFF_STAGE + (size_t)p.rank * 2 * NKS * A_BYTES) + (call & 1u) * (unsigned)SLOT_BYTES +
                                  (unsigned)(sq * 2048 + lane * 16);
            const unsigned foff0 = (unsigned)OFF_FLAGS + (unsigned)(p.rank * 2 * NKS * 16 + sq) * 4u;
            asm volatile("" :: "v"(d[0]), "v"(d[1]));
            unsigned raw = claim_issue(q, 1u);
            unsigned* cnt0 = (unsigned*)(p.ws + OFF_SCNT + (size_t)di * 64 * 64);
            const unsigned sfoff0 = (unsigned)OFF_FLAGS + (unsigned)(p.rank * 2 * NKS) * 4u;
            unsigned pflag = 0u, praw = 0u;
            while (true) {
                const unsigned st = cur >> 4, hf = (cur >> 3) & 1u, g2 = cur & 7u;
                const unsigned off = off0 + (hf * NKS + st) * (unsigned)A_BYTES + g2 * 4096u;
                push_1k(off, d[0], wsd);
                if (pflag != 0u) {
                    wait_vm<1>();
                    if ((claim_get<false>(praw) & 15u) == 15u) push_flag(pflag, call, wsd);
                }
                const unsigned pos = claim_get<true>(raw) + qoff;
                const bool more = pos < Q_ITEMS;
                const unsigned nxt = more ? pos : cur;
                v4i e[2];
                load_item(nxt, e);
                raw = claim_issue(q, more ? 1u : 0u);
                __builtin_amdgcn_s_sleep(PACE);
                push_1k(off + 1024u, d[1], wsd);
                wait_vm<0>();
                praw = claim_issue(cnt0 + (hf * NKS + st) * 16u, 1u);
                pflag = sfoff0 + (hf * NKS + st) * 4u;
                if (!more) break;
                __builtin_amdgcn_s_sleep(PACE);
                cur = nxt;
                d[0] = e[0]; d[1] = e[1];
            }
            if ((claim_get<true>(praw) & 15u) == 15u) push_flag(pflag, call, wsd);
        } else {
            __builtin_amdgcn_s_barrier();
        }
        wave_exit();
        return;
    }
    __builtin_amdgcn_s_barrier();
    asm volatile("" ::: "memory");
    const unsigned ctr_v = __hip_atomic_load(ctr, __ATOMIC_RELAXED, __HIP_MEMORY_SCOPE_AGENT);
    const unsigned call = (unsigned)__builtin_amdgcn_readfirstlane(ctr_v) / WGS + 1u;
    const unsigned slot = call & 1u;
#define STAMP(i)
    STAMP(0);

    const int mt = 4 * (xcd & 3) + (j & 3);
    const int nt = 8 * (xcd >> 2) + (j >> 2);
    unsigned* tmo = (unsigned*)(p.ws + OFF_TMO);

    const bool own = (mt >> 1) == p.rank;
    if (wave >= 8) {
        if (own) {
            const unsigned v = NKS;
            asm volatile("ds_write_b32 %0, %1" :: "v"(a_avail), "v"(v) : "memory");
        } else {
            const unsigned* f = (const unsigned*)(p.ws + OFF_FLAGS) + (size_t)mt * NKS + (lane & 31);
            const unsigned long long t0 = __builtin_amdgcn_s_memrealtime();
            unsigned av = 0u;
            for (unsigned it = 1; av < (unsigned)NKS; ++it) {
                const unsigned fv = __hip_atomic_load(f, __ATOMIC_RELAXED, __HIP_MEMORY_SCOPE_SYSTEM);
                const unsigned m = (unsigned)__builtin_amdgcn_ballot_w64(fv < call);
                unsigned v = m == 0u ? (unsigned)NKS : (unsigned)__builtin_ctz(m);
                if ((it & 15u) == 0u && v < (unsigned)NKS &&
                    (__hip_atomic_load(tmo, __ATOMIC_RELAXED, __HIP_MEMORY_SCOPE_AGENT) == call ||
                     __builtin_amdgcn_s_memrealtime() - t0 > DEADLINE_TICKS)) {
                    __hip_atomic_store(tmo, call, __ATOMIC_RELAXED, __HIP_MEMORY_SCOPE_AGENT);
                    asm volatile("s_waitcnt vmcnt(0)" ::: "memory");
                    v = NKS;
                }
                if (v > av) {
                    av = v;
                    asm volatile("ds_write_b32 %0, %1" :: "v"(a_avail), "v"(v) : "memory");
                    if (v < (unsigned)NKS) __builtin_amdgcn_s_sleep(POLL_GAP);
                } else {
                    __builtin_amdgcn_s_sleep(POLL_SLEEP);
                }
            }
        }
        wave_exit();
        return;
    }

    const unsigned char* a_src = p.ws + OFF_STAGE + slot * SLOT_BYTES + (size_t)mt * NKS * A_BYTES;
    const unsigned char* b_src = p.w + (size_t)p.rank * NS + nt * BN;

    const unsigned char* g_ptr;
    int g_odd, g_pair, g_step, l_off;
    if (wave < 4) {
        l_off = wave * 8192;
        if (own) {
            const int row = (mt & 1) * 256 + wave * 64 + (lane >> 3);
            const int ce = (lane & 7) ^ (lane >> 4), co = ce ^ 4;
            g_ptr = p.x + (size_t)row * KDIM + ce * 16;
            g_odd = 8 * KDIM + (co - ce) * 16; g_pair = 16 * KDIM; g_step = BK;
        } else {
            g_ptr = a_src + wave * 8192 + lane * 16;
            g_odd = 1024; g_pair = 2048; g_step = A_BYTES;
        }
    } else {
        const int wb = (wave - 4) & 1;
        const int kr = wb * 64 + (lane >> 3);
        const int ch = (lane & 7) ^ (((kr >> 1) & 3) << 1);
        g_ptr = b_src + (size_t)kr * NFULL + ch * 16;
        g_odd = 8 * NFULL; g_pair = 16 * NFULL; g_step = BK * NFULL; l_off = A_BYTES + wb * 8192;
    }
    auto issue_stage = [&](int s, int buf) {
        const unsigned char* g = g_ptr + (size_t)s * g_step;
        unsigned char* l = lds + buf * ST_BYTES + l_off;
        if (wave < 4) {
#pragma unroll
            for (int i = 0; i < 8; ++i)
                __builtin_amdgcn_global_load_lds((const unsigned*)(g + (i >> 1) * g_pair + (i & 1) * g_odd),
                                                 (unsigned*)(l + i * 1024), 16, 0, 17);
        } else if (wave < 6) {
#pragma unroll
            for (int i = 0; i < 8; ++i)
                __builtin_amdgcn_global_load_lds((const unsigned*)(g + (i >> 1) * g_pair + (i & 1) * g_odd),
                                                 (unsigned*)(l + i * 1024), 16, 0, 0);
        }
    };
    auto wait_avail = [&](int s) {
        const unsigned need = (unsigned)s + 1u;
        while (lds_ld(a_avail) < need) __builtin_amdgcn_s_sleep(1);
    };
    auto arrive = [&]() {
        if (lane == 0) {
            unsigned one = 1u;
            asm volatile("ds_add_u32 %0, %1" :: "v"(a_cnt), "v"(one) : "memory");
        }
    };

    const int wm = wave >> 1, wn = wave & 1;
    const int h = lane >> 5, r31 = lane & 31;
    const int ax = (r31 >> 1) & 7;
    const int a_base = (wm * 64 + r31) * 128;
    int a_off[2][2];
#pragma unroll
    for (int s = 0; s < 2; ++s)
#pragma unroll
        for (int e = 0; e < 2; ++e) a_off[s][e] = a_base + (((4 * s + 2 * h + e) ^ ax) << 4);
    const int g = lane >> 4, q = (lane & 15) >> 1, pp = lane & 1;
    int b_off[2];
#pragma unroll
    for (int nb = 0; nb < 2; ++nb)
        b_off[nb] = A_BYTES + (32 * h + q) * 128 + (((wn * 4 + nb * 2 + (g & 1)) ^ (((q >> 1) & 3) << 1)) << 4) + pp * 8;

    v16f acc[2][2];
#pragma unroll
    for (int i = 0; i < 2; ++i)
#pragma unroll
        for (int k = 0; k < 2; ++k)
#pragma unroll
            for (int r = 0; r < 16; ++r) acc[i][k][r] = 0.f;

    if (wave < 4) wait_avail(0);
    STAMP(1);
    issue_stage(0, 0);
    if (wave < 4) wait_avail(1);
    issue_stage(1, 1);
    wait_vm<8>();
    arrive();

    int buf = 0;
    for (int s = 0; s < NKS; ++s) {
        const int s2 = s + 2;
        int nbuf = buf + 2; if (nbuf >= NBUF) nbuf -= NBUF;
        {
            const unsigned target = 8u * (unsigned)(s + 1);
            while (lds_ld(a_cnt) < target) {}
        }
        bool issued = true;
        if (s2 < NKS) {
            if (wave < 4) issued = lds_ld(a_avail) >= (unsigned)s2 + 1u;
            if (issued) issue_stage(s2, nbuf);
        }

        const unsigned lb = lds_base + buf * ST_BYTES;
        v4i alo[2][2], ahi[2][2];
        v2i bt[2][2][4];
        {
            const unsigned a0 = lb + a_off[0][0], a1 = lb + a_off[0][1];
            const unsigned a2 = lb + a_off[1][0], a3 = lb + a_off[1][1];
            const unsigned b0 = lb + b_off[0], b1 = lb + b_off[1];
            DS_B128(alo[0][0], a0, 0);    DS_B128(ahi[0][0], a1, 0);
            DS_TR8(bt[0][0][0], b0, 0);   DS_TR8(bt[0][0][1], b0, 1024);
            DS_TR8(bt[0][0][2], b0, 2048); DS_TR8(bt[0][0][3], b0, 3072);
            DS_B128(alo[0][1], a0, 4096); DS_B128(ahi[0][1], a1, 4096);
            DS_TR8(bt[0][1][0], b1, 0);   DS_TR8(bt[0][1][1], b1, 1024);
            DS_TR8(bt[0][1][2], b1, 2048); DS_TR8(bt[0][1][3], b1, 3072);
            DS_B128(alo[1][0], a2, 0);    DS_B128(ahi[1][0], a3, 0);
            DS_TR8(bt[1][0][0], b0, 8192); DS_TR8(bt[1][0][1], b0, 9216);
            DS_TR8(bt[1][0][2], b0, 10240); DS_TR8(bt[1][0][3], b0, 11264);
            DS_B128(alo[1][1], a2, 4096); DS_B128(ahi[1][1], a3, 4096);
            DS_TR8(bt[1][1][0], b1, 8192); DS_TR8(bt[1][1][1], b1, 9216);
            DS_TR8(bt[1][1][2], b1, 10240); DS_TR8(bt[1][1][3], b1, 11264);
        }
#pragma unroll
        for (int ks = 0; ks < 2; ++ks) {
            if (ks == 0) {
                asm volatile("s_waitcnt lgkmcnt(12)" ::: "memory");
            } else {
                asm volatile("s_waitcnt lgkmcnt(0)" ::: "memory");
                if (s + 1 < NKS) {
                    if (s2 < NKS && issued) wait_vm<8>(); else wait_vm<0>();
                    arrive();
                }
            }
            __builtin_amdgcn_sched_barrier(0);
            v8i af[2], bf[2];
#pragma unroll
            for (int mb = 0; mb < 2; ++mb)
                af[mb] = __builtin_shufflevector(alo[ks][mb], ahi[ks][mb], 0, 1, 2, 3, 4, 5, 6, 7);
#pragma unroll
            for (int nb = 0; nb < 2; ++nb) {
                v4i lo = __builtin_shufflevector(bt[ks][nb][0], bt[ks][nb][1], 0, 1, 2, 3);
                v4i hi = __builtin_shufflevector(bt[ks][nb][2], bt[ks][nb][3], 0, 1, 2, 3);
                bf[nb] = __builtin_shufflevector(lo, hi, 0, 1, 2, 3, 4, 5, 6, 7);
            }
#pragma unroll
            for (int mb = 0; mb < 2; ++mb)
#pragma unroll
                for (int nb = 0; nb < 2; ++nb)
                    acc[mb][nb] = __builtin_amdgcn_mfma_scale_f32_32x32x64_f8f6f4(
                        af[mb], bf[nb], acc[mb][nb], 0, 0, 0, 0x7f7f7f7f, 0, 0x7f7f7f7f);
            __builtin_amdgcn_sched_barrier(0);
        }
        if (!issued) { wait_avail(s2); issue_stage(s2, nbuf); }
        buf = buf + 1; if (buf >= NBUF) buf = 0;
    }

    STAMP(2);
    const float scale = p.sx[0] * p.sw[0];
    const bool bad = __hip_atomic_load(tmo, __ATOMIC_RELAXED, __HIP_MEMORY_SCOPE_AGENT) == call;
    const float nanv = __builtin_nanf("");
    float* o = p.out + (size_t)(mt * BM + wm * 64 + 4 * h) * NS + nt * BN + wn * 64 + r31;
#pragma unroll
    for (int mb = 0; mb < 2; ++mb)
#pragma unroll
        for (int nb = 0; nb < 2; ++nb)
#pragma unroll
            for (int r = 0; r < 16; ++r) {
                const int row = mb * 32 + (r & 3) + 8 * (r >> 2);
                float v = fmaxf(acc[mb][nb][r] * scale, 0.f);
                __hip_atomic_store(&o[(size_t)row * NS + nb * 32], bad ? nanv : v, __ATOMIC_RELAXED, __HIP_MEMORY_SCOPE_AGENT);
            }
    STAMP(3);
    wave_exit();
}

}

extern "C" void kernel_launch(int rank, int nranks, ncclComm_t comm,
                              hipStream_t stream,
                              void* const* d_in, const int* in_sizes, int n_in,
                              void* d_out, int out_size,
                              const int* in_dtypes, int out_dtype,
                              void* const* d_out_all,
                              void* const* d_ws_all, size_t ws_bytes) {
    (void)comm; (void)in_sizes; (void)n_in; (void)out_size; (void)in_dtypes; (void)out_dtype; (void)d_out_all;
    if (nranks != NR || ws_bytes < WS_NEED || (unsigned)rank >= (unsigned)NR) {
        fprintf(stderr, "unexpected configuration\n");
        return;
    }
    HIP_CHECK(hipFuncSetAttribute((const void*)fused_kernel, hipFuncAttributeMaxDynamicSharedMemorySize, LDS_BYTES));
    Params p;
    p.x = (const unsigned char*)d_in[0];
    p.w = (const unsigned char*)d_in[1];
    p.sx = (const float*)d_in[2];
    p.sw = (const float*)d_in[3];
    p.out = (float*)d_out;
    p.ws_all = d_ws_all;
    p.ws = (unsigned char*)d_ws_all[rank];
    p.rank = rank;
    fused_kernel<<<WGS, THREADS, LDS_BYTES, stream>>>(p);
}
```

```cpp
#include <hip/hip_runtime.h>
#include <rccl/rccl.h>
#include <cstdio>
#include <cstdint>

#define HIP_CHECK(expr) do { hipError_t _e = (expr); if (_e != hipSuccess) { \
    fprintf(stderr, "HIP error %s at %s:%d\n", hipGetErrorString(_e), __FILE__, __LINE__); \
    return; } } while (0)

namespace {

typedef int   v8i  __attribute__((ext_vector_type(8)));
typedef int   v4i  __attribute__((ext_vector_type(4)));
typedef int   v2i  __attribute__((ext_vector_type(2)));
typedef float v16f __attribute__((ext_vector_type(16)));

constexpr int NR    = 8;
constexpr int KDIM  = 4096;
constexpr int NFULL = 16384;
constexpr int NS    = 2048;
constexpr int BM = 256, BN = 128, BK = 128;
constexpr int NKS   = KDIM / BK;
#define CHST 2
constexpr int CH_ST = CHST;
constexpr int NCH   = NKS / CH_ST;
constexpr unsigned Q_ITEMS = NCH * 32;
constexpr unsigned Q_WAVES = 32;
#define PACE 6
#define AGG
#define POLL_GAP 20
#define POLL_SLEEP 6
#define POLL_HGAP POLL_SLEEP
#define LEAD 3u
#define SEND_FIRST
#define INIT_TID 128
#define LATE_CLAIM
#define PRIO_S 3
#define NO_PIN
#define AV_SLEEP 1
#define ZORD 0
#define A_AUX 17
#define B_AUX 0
constexpr int A_BYTES = BM * BK;
constexpr int B_BYTES = BK * BN;
constexpr int ST_BYTES = A_BYTES + B_BYTES;
constexpr int NBUF = 3;
constexpr int SYNC_OFF = NBUF * ST_BYTES;
constexpr int LDS_BYTES = SYNC_OFF + 64;

constexpr int WGS = 256, THREADS = 704;
#define ST_BITS " sc0 sc1"
#define LAG 0
#define W1

constexpr size_t OFF_CTR   = 0;
constexpr size_t OFF_TMO   = 512;
constexpr size_t OFF_QUEUE = 4096;
constexpr size_t OFF_FLAGS = 8192;
constexpr size_t OFF_FLAGS2 = 16384;
constexpr size_t OFF_SCNT  = 65536;
constexpr size_t OFF_STAGE = 1u << 20;
constexpr size_t SLOT_BYTES = (size_t)NR * 2 * NKS * A_BYTES;
constexpr size_t WS_NEED   = OFF_STAGE + 2 * SLOT_BYTES;

constexpr unsigned long long DEADLINE_TICKS = 150000000ull;

template <int N> __device__ __forceinline__ void wait_vm() {
    asm volatile("s_waitcnt vmcnt(%0)" :: "n"(N) : "memory");
}
#define DS_B128(dst, addr, off) asm volatile("ds_read_b128 %0, %1 offset:" #off : "=v"(dst) : "v"(addr) : "memory")
#define DS_TR8(dst, addr, off)  asm volatile("ds_read_b64_tr_b8 %0, %1 offset:" #off : "=v"(dst) : "v"(addr) : "memory")

__device__ __forceinline__ unsigned lds_ld(unsigned addr) {
    unsigned v;
    asm volatile("ds_read_b32 %0, %1\n\ts_waitcnt lgkmcnt(0)" : "=v"(v) : "v"(addr) : "memory");
    return (unsigned)__builtin_amdgcn_readfirstlane(v);
}

struct Params {
    const unsigned char* x;
    const unsigned char* w;
    const float* sx;
    const float* sw;
    float* out;
    unsigned char* ws;
    unsigned char* peer_ws[NR];
    int rank;
};

__device__ __forceinline__ unsigned claim_issue(unsigned* q, unsigned inc) {
    unsigned r;
    const unsigned zero = 0u;
    asm volatile("s_mov_b64 exec, 1\n\tglobal_atomic_add %0, %1, %2, %3 sc0\n\ts_mov_b64 exec, -1"
                 : "=&v"(r) : "v"(zero), "v"(inc), "s"(q) : "memory");
    return r;
}
template <bool WAIT> __device__ __forceinline__ unsigned claim_get(unsigned r) {
    unsigned v;
    if constexpr (WAIT) asm volatile("s_waitcnt vmcnt(0)\n\tv_readfirstlane_b32 %0, %1\n\ts_nop 4" : "=s"(v) : "v"(r) : "memory");
    else                asm volatile("v_readfirstlane_b32 %0, %1\n\ts_nop 4" : "=s"(v) : "v"(r) : "memory");
    return v;
}
__device__ __forceinline__ void push_1k(unsigned off, v4i d, const unsigned char* base) {
    asm volatile("global_store_dwordx4 %0, %1, %2 sc0 sc1\n\ts_nop 1" :: "v"(off), "v"(d), "s"(base) : "memory");
}
__device__ __forceinline__ void push_flag(unsigned off, unsigned v, const unsigned char* base) {
    asm volatile("global_store_dword %0, %1, %2 sc0 sc1\n\ts_nop 1" :: "v"(off), "v"(v), "s"(base) : "memory");
}

__global__ void __launch_bounds__(THREADS, 3) fused_kernel(Params pa) {
    extern __shared__ __attribute__((aligned(16))) unsigned char lds[];
    const int tid  = threadIdx.x;
    const int lane = tid & 63;
    const int wave_hw = __builtin_amdgcn_readfirstlane(tid >> 6);
    const int wave = wave_hw < 3 ? wave_hw + 8 : wave_hw - 3;
    const int b = blockIdx.x;
    struct { const unsigned char* x; const unsigned char* w; const float* sx; const float* sw; float* out;
             unsigned char* ws; unsigned char* wsd; int rank; } p;
    {
        typedef __attribute__((address_space(1))) unsigned char gu8;
        gu8* gx = (gu8*)pa.x; gu8* gw = (gu8*)pa.w; gu8* gsx = (gu8*)pa.sx; gu8* gsw = (gu8*)pa.sw;
        gu8* go = (gu8*)pa.out; gu8* gws = (gu8*)pa.ws; gu8* gwsd = (gu8*)pa.peer_ws[(unsigned)((b >> 3) + b) & 7u];
        int rk = pa.rank;
        asm volatile("" : "+s"(gx), "+s"(gw), "+s"(gsx), "+s"(gsw), "+s"(go), "+s"(gws), "+s"(gwsd), "+s"(rk));
        p.x = (const unsigned char*)gx; p.w = (const unsigned char*)gw; p.sx = (const float*)gsx; p.sw = (const float*)gsw;
        p.out = (float*)go; p.ws = (unsigned char*)gws; p.wsd = (unsigned char*)gwsd; p.rank = rk;
    }
    const unsigned lds_base = (unsigned)(size_t)(__attribute__((address_space(3))) unsigned char*)lds;
    const unsigned a_cnt = lds_base + SYNC_OFF, a_avail = lds_base + SYNC_OFF + 16;

    const int xcd = b & 7, j = b >> 3;
    unsigned* ctr = (unsigned*)(p.ws + OFF_CTR);
    const unsigned a_exit = lds_base + SYNC_OFF + 32;
    if (tid == INIT_TID) {
        const unsigned z = 0u;
        asm volatile("ds_write_b32 %0, %1\n\tds_write_b32 %0, %1 offset:16\n\tds_write_b32 %0, %1 offset:32\n\t"
                     "ds_write_b32 %0, %1 offset:48\n\tds_write_b32 %0, %1 offset:4\n\ts_waitcnt lgkmcnt(0)" :: "v"(a_cnt), "v"(z) : "memory");
    }
    auto wave_exit = [&]() {
        if (lane == 0) {
            unsigned old;
            const unsigned one = 1u;
            asm volatile("ds_add_rtn_u32 %0, %1, %2\n\ts_waitcnt lgkmcnt(0)" : "=v"(old) : "v"(a_exit), "v"(one) : "memory");
            if (old == (unsigned)(THREADS / 64 - 1))
                __hip_atomic_fetch_add(ctr, 1u, __ATOMIC_RELAXED, __HIP_MEMORY_SCOPE_AGENT);
        }
    };
    if (wave >= 8) __builtin_amdgcn_s_setprio(PRIO_S);
    if (wave == 8 || wave == 9) {
        unsigned ctr_v = __hip_atomic_load(ctr, __ATOMIC_RELAXED, __HIP_MEMORY_SCOPE_AGENT);
        const int di = (j + xcd) & 7;
        if (di < NR - 1) {
            const int sq = wave & 1;
            unsigned* q = (unsigned*)(p.ws + OFF_QUEUE + (size_t)(di * 2 + sq) * 256);
            const unsigned char* wsd = p.wsd;
            const unsigned lsrc = (unsigned)((sq * 16 + (lane >> 3)) * KDIM + (((lane & 7) ^ (lane >> 4)) << 4));
            auto remap = [&](unsigned pos) -> unsigned {
                const unsigned gi = pos >> 3, t = gi - LEAD;
                unsigned st, hf;
                if (gi < LEAD) { hf = 0u; st = gi; }
                else if (gi < 64u - LEAD) { hf = (t & 1u) ^ 1u; st = (t >> 1) + ((t & 1u) ? LEAD : 0u); }
                else { hf = 1u; st = gi - 32u; }
                return (st << 4) | (hf << 3) | (pos & 7u);
            };
            auto load_item = [&](unsigned it, v4i* dst) {
                const unsigned char* sp = p.x + (size_t)((it & 15u) * (32u * KDIM) + (it >> 4) * BK);
                dst[0] = *(const v4i*)(sp + lsrc);
                dst[1] = *(const v4i*)(sp + 8 * KDIM + (lsrc ^ 64u));
            };
            unsigned cur = remap((unsigned)(xcd * 4 + (j >> 3)));
            v4i d[2];
            load_item(cur, d);
            __builtin_amdgcn_s_barrier();
            asm volatile("" : "+v"(ctr_v));
            const unsigned call = (unsigned)__builtin_amdgcn_readfirstlane(ctr_v) / WGS + 1u;
            const unsigned qoff = Q_WAVES - (call - 1u) * Q_ITEMS;
            const unsigned off0 = (unsigned)(OFF_STAGE + (size_t)p.rank * 2 * NKS * A_BYTES) + (call & 1u) * (unsigned)SLOT_BYTES +
                                  (unsigned)(sq * 2048 + lane * 16);
            const unsigned foff0 = (unsigned)OFF_FLAGS + (unsigned)(p.rank * 2 * NKS * 16 + sq) * 4u;
            asm volatile("" :: "v"(d[0]), "v"(d[1]));
            unsigned* cnt0 = (unsigned*)(p.ws + OFF_SCNT + (size_t)di * 64 * 64);
            const unsigned sfoff0 = (unsigned)OFF_FLAGS + (unsigned)(p.rank * 2 * NKS) * 4u;
            unsigned pflag = 0u, praw = 0u;
            while (true) {
                const unsigned st = cur >> 4, hf = (cur >> 3) & 1u, g2 = cur & 7u;
                const unsigned off = off0 + (hf * NKS + st) * (unsigned)A_BYTES + g2 * 4096u;
                push_1k(off, d[0], wsd);
                const unsigned raw = claim_issue(q, 1u);
                if (pflag != 0u) {
                    wait_vm<2>();
                    if ((claim_get<false>(praw) & 15u) == 15u) push_flag(pflag, call, wsd);
                }
                const unsigned pos = claim_get<true>(raw) + qoff;
                const bool more = pos < Q_ITEMS;
                const unsigned nxt = more ? remap(pos) : cur;
                v4i e[2];
                load_item(nxt, e);
                __builtin_amdgcn_s_sleep(PACE);
                push_1k(off + 1024u, d[1], wsd);
                wait_vm<0>();
                {
                    praw = claim_issue(cnt0 + (hf * NKS + st) * 16u, 1u);
                    pflag = sfoff0 + (hf * NKS + st) * 4u;
                }
                if (!more) break;
                __builtin_amdgcn_s_sleep(PACE);
                cur = nxt;
                d[0] = e[0]; d[1] = e[1];
            }
            if (pflag != 0u && (claim_get<true>(praw) & 15u) == 15u) push_flag(pflag, call, wsd);
        } else {
            __builtin_amdgcn_s_barrier();
        }
        wave_exit();
        return;
    }
    __builtin_amdgcn_s_barrier();
    asm volatile("" ::: "memory");
    const unsigned ctr_v = __hip_atomic_load(ctr, __ATOMIC_RELAXED, __HIP_MEMORY_SCOPE_AGENT);
    int scale_i = __builtin_amdgcn_readfirstlane(__float_as_int(p.sx[0] * p.sw[0]));
    asm volatile("" : "+s"(scale_i));
    const unsigned call = (unsigned)__builtin_amdgcn_readfirstlane(ctr_v) / WGS + 1u;
    const unsigned slot = call & 1u;
#define STAMP(i)
    STAMP(0);

    const int mt = 4 * (xcd & 3) + (j & 3);
    const int nt = 8 * (xcd >> 2) + (j >> 2);
    unsigned* tmo = (unsigned*)(p.ws + OFF_TMO);

    const bool own = (mt >> 1) == p.rank;
    if (wave >= 8) {
        if (own) {
            const unsigned v = NKS;
            asm volatile("ds_write_b32 %0, %1" :: "v"(a_avail), "v"(v) : "memory");
        } else {
            const unsigned* f = (const unsigned*)(p.ws + OFF_FLAGS) + (size_t)mt * NKS + (lane & 31);
            const unsigned long long t0 = __builtin_amdgcn_s_memrealtime();
            unsigned av = 0u;
            for (unsigned it = 1; av < (unsigned)NKS; ++it) {
                const unsigned fv = __hip_atomic_load(f, __ATOMIC_RELAXED, __HIP_MEMORY_SCOPE_SYSTEM);
                const unsigned m = (unsigned)__builtin_amdgcn_ballot_w64(fv < call);
                unsigned v = m == 0u ? (unsigned)NKS : (unsigned)__builtin_ctz(m);
                if ((it & 15u) == 0u && v < (unsigned)NKS &&
                    (__hip_atomic_load(tmo, __ATOMIC_RELAXED, __HIP_MEMORY_SCOPE_AGENT) == call ||
                     __builtin_amdgcn_s_memrealtime() - t0 > DEADLINE_TICKS)) {
                    __hip_atomic_store(tmo, call, __ATOMIC_RELAXED, __HIP_MEMORY_SCOPE_AGENT);
                    asm volatile("ds_write_b32 %0, %1 offset:32" :: "v"(a_avail), "v"(call) : "memory");
                    v = NKS;
                }
                if (v > av) {
                    av = v;
                    asm volatile("ds_write_b32 %0, %1" :: "v"(a_avail), "v"(v) : "memory");
                    if (v < (unsigned)NKS) __builtin_amdgcn_s_sleep(POLL_GAP);
                } else {
                    __builtin_amdgcn_s_sleep(POLL_SLEEP);
                }
            }
        }
        wave_exit();
        return;
    }

    const unsigned char* a_src = p.ws + OFF_STAGE + slot * SLOT_BYTES + (size_t)mt * NKS * A_BYTES;
    const unsigned char* b_src = p.w + (size_t)p.rank * NS + nt * BN;

    const unsigned char* g_ptr;
    int g_odd, g_pair, g_step, l_off;
    if (wave < 4) {
        l_off = wave * 8192;
        if (own) {
            const int row = (mt & 1) * 256 + wave * 64 + (lane >> 3);
            const int ce = (lane & 7) ^ (lane >> 4), co = ce ^ 4;
            g_ptr = p.x + (size_t)row * KDIM + ce * 16;
            g_odd = 8 * KDIM + (co - ce) * 16; g_pair = 16 * KDIM; g_step = BK;
        } else {
            g_ptr = a_src + wave * 8192 + lane * 16;
            g_odd = 1024; g_pair = 2048; g_step = A_BYTES;
        }
    } else {
        const int wb = (wave - 4) & 1;
        const int kr = wb * 64 + (lane >> 3);
        l_off = A_BYTES + wb * 8192;
        const int ch = (lane & 7) ^ (((kr >> 1) & 3) << 1);
        g_ptr = b_src + (size_t)kr * NFULL + ch * 16;
        g_odd = 8 * NFULL; g_pair = 16 * NFULL; g_step = BK * NFULL;
    }
    auto issue_stage = [&](int s, int buf) {
        const unsigned char* g = g_ptr + (size_t)s * g_step;
        unsigned char* l = lds + buf * ST_BYTES + l_off;
        if (wave < 4) {
#pragma unroll
            for (int i = 0; i < 8; ++i)
                __builtin_amdgcn_global_load_lds((const unsigned*)(g + (i >> 1) * g_pair + (i & 1) * g_odd),
                                                 (unsigned*)(l + i * 1024), 16, 0, A_AUX);
        } else if (wave < 6) {
#pragma unroll
            for (int i = 0; i < 8; ++i)
                __builtin_amdgcn_global_load_lds((const unsigned*)(g + (i >> 1) * g_pair + (i & 1) * g_odd),
                                                 (unsigned*)(l + i * 1024), 16, 0, B_AUX);
        }
    };
    auto wait_avail = [&](int s) {
        const unsigned need = (unsigned)s + 1u;
        while (lds_ld(a_avail) < need) __builtin_amdgcn_s_sleep(AV_SLEEP);
    };
    auto wait_older = [&]() {
        wait_vm<8>();
    };
    auto arrive = [&]() {
        if (lane == 0) {
            unsigned one = 1u;
            asm volatile("ds_add_u32 %0, %1" :: "v"(a_cnt), "v"(one) : "memory");
        }
    };

    const int wm = wave >> 1, wn = wave & 1;
    const int h = lane >> 5, r31 = lane & 31;
    const int ax = (r31 >> 1) & 7;
    const int a_base = (wm * 64 + r31) * 128;
    int a_off[2][2];
#pragma unroll
    for (int s = 0; s < 2; ++s)
#pragma unroll
        for (int e = 0; e < 2; ++e) a_off[s][e] = a_base + (((4 * s + 2 * h + e) ^ ax) << 4);
    const int g = lane >> 4, q = (lane & 15) >> 1, pp = lane & 1;
    int b_off[2];
#pragma unroll
    for (int nb = 0; nb < 2; ++nb)
        b_off[nb] = A_BYTES + (32 * h + q) * 128 + (((wn * 4 + nb * 2 + (g & 1)) ^ (((q >> 1) & 3) << 1)) << 4) + pp * 8;

    v16f acc[2][2];
#pragma unroll
    for (int i = 0; i < 2; ++i)
#pragma unroll
        for (int k = 0; k < 2; ++k)
#pragma unroll
            for (int r = 0; r < 16; ++r) acc[i][k][r] = 0.f;

    if (wave < 4) wait_avail(0);
    STAMP(1);
    issue_stage(0, 0);
    if (wave < 4) wait_avail(1);
    issue_stage(1, 1);
    wait_older();
    arrive();

    int buf = 0;
    for (int s = 0; s < NKS; ++s) {
        const int s2 = s + 2;
        int nbuf = buf + 2; if (nbuf >= NBUF) nbuf -= NBUF;
        {
            const unsigned target = 8u * (unsigned)(s + 1);
            while (lds_ld(a_cnt) < target) {}
        }
        bool issued = true;
        if (s2 < NKS) {
            if (wave < 4) issued = lds_ld(a_avail) >= (unsigned)s2 + 1u;
            if (issued) issue_stage(s2, nbuf);
        }

        const unsigned lb = lds_base + buf * ST_BYTES;
        v4i alo[2][2], ahi[2][2];
        v2i bt[2][2][4];
        {
            const unsigned a0 = lb + a_off[0][0], a1 = lb + a_off[0][1];
            const unsigned a2 = lb + a_off[1][0], a3 = lb + a_off[1][1];
            const unsigned b0 = lb + b_off[0], b1 = lb + b_off[1];
            DS_B128(alo[0][0], a0, 0);    DS_B128(ahi[0][0], a1, 0);
            DS_TR8(bt[0][0][0], b0, 0);   DS_TR8(bt[0][0][1], b0, 1024);
            DS_TR8(bt[0][0][2], b0, 2048); DS_TR8(bt[0][0][3], b0, 3072);
            DS_B128(alo[0][1], a0, 4096); DS_B128(ahi[0][1], a1, 4096);
            DS_TR8(bt[0][1][0], b1, 0);   DS_TR8(bt[0][1][1], b1, 1024);
            DS_TR8(bt[0][1][2], b1, 2048); DS_TR8(bt[0][1][3], b1, 3072);
            DS_B128(alo[1][0], a2, 0);    DS_B128(ahi[1][0], a3, 0);
            DS_TR8(bt[1][0][0], b0, 8192); DS_TR8(bt[1][0][1], b0, 9216);
            DS_TR8(bt[1][0][2], b0, 10240); DS_TR8(bt[1][0][3], b0, 11264);
            DS_B128(alo[1][1], a2, 4096); DS_B128(ahi[1][1], a3, 4096);
            DS_TR8(bt[1][1][0], b1, 8192); DS_TR8(bt[1][1][1], b1, 9216);
            DS_TR8(bt[1][1][2], b1, 10240); DS_TR8(bt[1][1][3], b1, 11264);
        }
#pragma unroll
        for (int ks = 0; ks < 2; ++ks) {
            if (ks == 0) {
                asm volatile("s_waitcnt lgkmcnt(12)" ::: "memory");
            } else {
                asm volatile("s_waitcnt lgkmcnt(0)" ::: "memory");
                if (s + 1 < NKS) {
                    if (s2 < NKS && issued) wait_older(); else wait_vm<0>();
                    arrive();
                }
            }
            __builtin_amdgcn_sched_barrier(0);
            v8i af[2], bf[2];
#pragma unroll
            for (int mb = 0; mb < 2; ++mb)
                af[mb] = __builtin_shufflevector(alo[ks][mb], ahi[ks][mb], 0, 1, 2, 3, 4, 5, 6, 7);
#pragma unroll
            for (int nb = 0; nb < 2; ++nb) {
                v4i lo = __builtin_shufflevector(bt[ks][nb][0], bt[ks][nb][1], 0, 1, 2, 3);
                v4i hi = __builtin_shufflevector(bt[ks][nb][2], bt[ks][nb][3], 0, 1, 2, 3);
                bf[nb] = __builtin_shufflevector(lo, hi, 0, 1, 2, 3, 4, 5, 6, 7);
            }
#pragma unroll
            for (int mb = 0; mb < 2; ++mb)
#pragma unroll
                for (int nb = 0; nb < 2; ++nb)
                    acc[mb][nb] = __builtin_amdgcn_mfma_scale_f32_32x32x64_f8f6f4(
                        af[mb], bf[nb], acc[mb][nb], 0, 0, 0, 0x7f7f7f7f, 0, 0x7f7f7f7f);
            __builtin_amdgcn_sched_barrier(0);
        }
        if (!issued) { wait_avail(s2); issue_stage(s2, nbuf); }
        buf = buf + 1; if (buf >= NBUF) buf = 0;
    }

    STAMP(2);
    const bool bad = lds_ld(a_avail + 32) != 0u;
    const float scale = __int_as_float(scale_i);
    const float nanv = __builtin_nanf("");
    float* o = p.out + (size_t)(mt * BM + wm * 64 + 4 * h) * NS + nt * BN + wn * 64 + r31;
#pragma unroll
    for (int mb = 0; mb < 2; ++mb)
#pragma unroll
        for (int nb = 0; nb < 2; ++nb)
#pragma unroll
            for (int r = 0; r < 16; ++r) {
                const int row = mb * 32 + (r & 3) + 8 * (r >> 2);
                float v = fmaxf(acc[mb][nb][r] * scale, 0.f);
                __hip_atomic_store(&o[(size_t)row * NS + nb * 32], bad ? nanv : v, __ATOMIC_RELAXED, __HIP_MEMORY_SCOPE_AGENT);
            }
    STAMP(3);
    wave_exit();
}

}

extern "C" void kernel_launch(int rank, int nranks, ncclComm_t comm,
                              hipStream_t stream,
                              void* const* d_in, const int* in_sizes, int n_in,
                              void* d_out, int out_size,
                              const int* in_dtypes, int out_dtype,
                              void* const* d_out_all,
                              void* const* d_ws_all, size_t ws_bytes) {
    (void)comm; (void)in_sizes; (void)n_in; (void)out_size; (void)in_dtypes; (void)out_dtype; (void)d_out_all;
    if (nranks != NR || ws_bytes < WS_NEED || (unsigned)rank >= (unsigned)NR) {
        fprintf(stderr, "unexpected configuration\n");
        return;
    }
    HIP_CHECK(hipFuncSetAttribute((const void*)fused_kernel, hipFuncAttributeMaxDynamicSharedMemorySize, LDS_BYTES));
    Params p;
    p.x = (const unsigned char*)d_in[0];
    p.w = (const unsigned char*)d_in[1];
    p.sx = (const float*)d_in[2];
    p.sw = (const float*)d_in[3];
    p.out = (float*)d_out;
    p.ws = (unsigned char*)d_ws_all[rank];
    for (int i = 0; i < NR; ++i) p.peer_ws[i] = (unsigned char*)d_ws_all[(unsigned)(rank + 1 + i) & 7u];
    p.rank = rank;
    fused_kernel<<<WGS, THREADS, LDS_BYTES, stream>>>(p);
}
```

```cpp
#include <hip/hip_runtime.h>
#include <rccl/rccl.h>
#include <cstdio>
#include <cstdint>

#define HIP_CHECK(expr) do { hipError_t _e = (expr); if (_e != hipSuccess) { \
    fprintf(stderr, "HIP error %s at %s:%d\n", hipGetErrorString(_e), __FILE__, __LINE__); \
    return; } } while (0)

namespace {

typedef int   v8i  __attribute__((ext_vector_type(8)));
typedef int   v4i  __attribute__((ext_vector_type(4)));
typedef int   v2i  __attribute__((ext_vector_type(2)));
typedef float v16f __attribute__((ext_vector_type(16)));

constexpr int NR    = 8;
constexpr int KDIM  = 4096;
constexpr int NFULL = 16384;
constexpr int NS    = 2048;
constexpr int BM = 256, BN = 128, BK = 128;
constexpr int NKS   = KDIM / BK;
#define CHST 2
constexpr int CH_ST = CHST;
constexpr int NCH   = NKS / CH_ST;
constexpr unsigned Q_ITEMS = NCH * 32;
constexpr unsigned Q_WAVES = 32;
#define PACE 6
#define AGG
#define POLL_GAP 20
#define POLL_SLEEP 6
#define POLL_HGAP POLL_SLEEP
#define LEAD 2u
#define VMPOLL 0
#define VMFROM 26
#define SEND_FIRST
#define INIT_TID 128
#define LATE_CLAIM
#define PRIO_S 3
#define NO_PIN
#define AV_SLEEP 1
#define ZORD 0
#define MAP 0
#define A_AUX 17
#define B_AUX 0
constexpr int A_BYTES = BM * BK;
constexpr int B_BYTES = BK * BN;
constexpr int ST_BYTES = A_BYTES + B_BYTES;
constexpr int NBUF = 3;
constexpr int SYNC_OFF = NBUF * ST_BYTES;
constexpr int LDS_BYTES = SYNC_OFF + 64;

constexpr int WGS = 256, THREADS = 704;
#define ST_BITS " sc0 sc1"
#define LAG 0
#define W1

constexpr size_t OFF_CTR   = 0;
constexpr size_t OFF_TMO   = 512;
constexpr size_t OFF_QUEUE = 4096;
constexpr size_t OFF_FLAGS = 8192;
constexpr size_t OFF_FLAGS2 = 16384;
constexpr size_t OFF_SCNT  = 65536;
constexpr size_t OFF_STAGE = 1u << 20;
constexpr size_t SLOT_BYTES = (size_t)NR * 2 * NKS * A_BYTES;
constexpr size_t WS_NEED   = OFF_STAGE + 2 * SLOT_BYTES;

constexpr unsigned long long DEADLINE_TICKS = 150000000ull;

template <int N> __device__ __forceinline__ void wait_vm() {
    asm volatile("s_waitcnt vmcnt(%0)" :: "n"(N) : "memory");
}
#define DS_B128(dst, addr, off) asm volatile("ds_read_b128 %0, %1 offset:" #off : "=v"(dst) : "v"(addr) : "memory")
#define DS_TR8(dst, addr, off)  asm volatile("ds_read_b64_tr_b8 %0, %1 offset:" #off : "=v"(dst) : "v"(addr) : "memory")

__device__ __forceinline__ unsigned lds_ld(unsigned addr) {
    unsigned v;
    asm volatile("ds_read_b32 %0, %1\n\ts_waitcnt lgkmcnt(0)" : "=v"(v) : "v"(addr) : "memory");
    return (unsigned)__builtin_amdgcn_readfirstlane(v);
}

struct Params {
    const unsigned char* x;
    const unsigned char* w;
    const float* sx;
    const float* sw;
    float* out;
    unsigned char* ws;
    unsigned char* peer_ws[NR];
    int rank;
};

__device__ __forceinline__ unsigned claim_issue(unsigned* q, unsigned inc) {
    unsigned r;
    const unsigned zero = 0u;
    asm volatile("s_mov_b64 exec, 1\n\tglobal_atomic_add %0, %1, %2, %3 sc0\n\ts_mov_b64 exec, -1"
                 : "=&v"(r) : "v"(zero), "v"(inc), "s"(q) : "memory");
    return r;
}
template <bool WAIT> __device__ __forceinline__ unsigned claim_get(unsigned r) {
    unsigned v;
    if constexpr (WAIT) asm volatile("s_waitcnt vmcnt(0)\n\tv_readfirstlane_b32 %0, %1\n\ts_nop 4" : "=s"(v) : "v"(r) : "memory");
    else                asm volatile("v_readfirstlane_b32 %0, %1\n\ts_nop 4" : "=s"(v) : "v"(r) : "memory");
    return v;
}
__device__ __forceinline__ void push_1k(unsigned off, v4i d, const unsigned char* base) {
    asm volatile("global_store_dwordx4 %0, %1, %2 sc0 sc1\n\ts_nop 1" :: "v"(off), "v"(d), "s"(base) : "memory");
}
__device__ __forceinline__ void push_flag(unsigned off, unsigned v, const unsigned char* base) {
    asm volatile("global_store_dword %0, %1, %2 sc0 sc1\n\ts_nop 1" :: "v"(off), "v"(v), "s"(base) : "memory");
}

__global__ void __launch_bounds__(THREADS, 3) fused_kernel(Params pa) {
    extern __shared__ __attribute__((aligned(16))) unsigned char lds[];
    const int tid  = threadIdx.x;
    const int lane = tid & 63;
    const int wave_hw = __builtin_amdgcn_readfirstlane(tid >> 6);
    const int wave = wave_hw < 3 ? wave_hw + 8 : wave_hw - 3;
    const int b = blockIdx.x;
    struct { const unsigned char* x; const unsigned char* w; const float* sx; const float* sw; float* out;
             unsigned char* ws; unsigned char* wsd; int rank; } p;
    {
        typedef __attribute__((address_space(1))) unsigned char gu8;
        gu8* gx = (gu8*)pa.x; gu8* gw = (gu8*)pa.w; gu8* gsx = (gu8*)pa.sx; gu8* gsw = (gu8*)pa.sw;
        gu8* go = (gu8*)pa.out; gu8* gws = (gu8*)pa.ws; gu8* gwsd = (gu8*)pa.peer_ws[(unsigned)((b >> 3) + b) & 7u];
        int rk = pa.rank;
        asm volatile("" : "+s"(gx), "+s"(gw), "+s"(gsx), "+s"(gsw), "+s"(go), "+s"(gws), "+s"(gwsd), "+s"(rk));
        p.x = (const unsigned char*)gx; p.w = (const unsigned char*)gw; p.sx = (const float*)gsx; p.sw = (const float*)gsw;
        p.out = (float*)go; p.ws = (unsigned char*)gws; p.wsd = (unsigned char*)gwsd; p.rank = rk;
    }
    const unsigned lds_base = (unsigned)(size_t)(__attribute__((address_space(3))) unsigned char*)lds;
    const unsigned a_cnt = lds_base + SYNC_OFF, a_avail = lds_base + SYNC_OFF + 16;

    const int xcd = b & 7, j = b >> 3;
    unsigned* ctr = (unsigned*)(p.ws + OFF_CTR);
    const unsigned a_exit = lds_base + SYNC_OFF + 32;
    if (tid == INIT_TID) {
        const unsigned z = 0u;
        asm volatile("ds_write_b32 %0, %1\n\tds_write_b32 %0, %1 offset:16\n\tds_write_b32 %0, %1 offset:32\n\t"
                     "ds_write_b32 %0, %1 offset:48\n\tds_write_b32 %0, %1 offset:4\n\ts_waitcnt lgkmcnt(0)" :: "v"(a_cnt), "v"(z) : "memory");
    }
    auto wave_exit = [&]() {
        if (lane == 0) {
            unsigned old;
            const unsigned one = 1u;
            asm volatile("ds_add_rtn_u32 %0, %1, %2\n\ts_waitcnt lgkmcnt(0)" : "=v"(old) : "v"(a_exit), "v"(one) : "memory");
            if (old == (unsigned)(THREADS / 64 - 1))
                __hip_atomic_fetch_add(ctr, 1u, __ATOMIC_RELAXED, __HIP_MEMORY_SCOPE_AGENT);
        }
    };
    if (wave >= 8) __builtin_amdgcn_s_setprio(PRIO_S);
    if (wave == 8 || wave == 9) {
        unsigned ctr_v = __hip_atomic_load(ctr, __ATOMIC_RELAXED, __HIP_MEMORY_SCOPE_AGENT);
        const int di = (j + xcd) & 7;
        if (di < NR - 1) {
            const int sq = wave & 1;
            unsigned* q = (unsigned*)(p.ws + OFF_QUEUE + (size_t)(di * 2 + sq) * 256);
            const unsigned char* wsd = p.wsd;
            const unsigned lsrc = (unsigned)((sq * 16 + (lane >> 3)) * KDIM + (((lane & 7) ^ (lane >> 4)) << 4));
            auto remap = [&](unsigned pos) -> unsigned {
                const unsigned gi = pos >> 3, t = gi - LEAD;
                unsigned st, hf;
                if (gi < LEAD) { hf = 0u; st = gi; }
                else if (gi < 64u - LEAD) { hf = (t & 1u) ^ 1u; st = (t >> 1) + ((t & 1u) ? LEAD : 0u); }
                else { hf = 1u; st = gi - 32u; }
                return (st << 4) | (hf << 3) | (pos & 7u);
            };
            auto load_item = [&](unsigned it, v4i* dst) {
                const unsigned char* sp = p.x + (size_t)((it & 15u) * (32u * KDIM) + (it >> 4) * BK);
                dst[0] = *(const v4i*)(sp + lsrc);
                dst[1] = *(const v4i*)(sp + 8 * KDIM + (lsrc ^ 64u));
            };
            unsigned cur = remap((unsigned)(xcd * 4 + (j >> 3)));
            v4i d[2];
            load_item(cur, d);
            __builtin_amdgcn_s_barrier();
            asm volatile("" : "+v"(ctr_v));
            const unsigned call = (unsigned)__builtin_amdgcn_readfirstlane(ctr_v) / WGS + 1u;
            const unsigned qoff = Q_WAVES - (call - 1u) * Q_ITEMS;
            const unsigned off0 = (unsigned)(OFF_STAGE + (size_t)p.rank * 2 * NKS * A_BYTES) + (call & 1u) * (unsigned)SLOT_BYTES +
                                  (unsigned)(sq * 2048 + lane * 16);
            const unsigned foff0 = (unsigned)OFF_FLAGS + (unsigned)(p.rank * 2 * NKS * 16 + sq) * 4u;
            asm volatile("" :: "v"(d[0]), "v"(d[1]));
            unsigned* cnt0 = (unsigned*)(p.ws + OFF_SCNT + (size_t)di * 64 * 64);
            const unsigned sfoff0 = (unsigned)OFF_FLAGS + (unsigned)(p.rank * 2 * NKS) * 4u;
            unsigned pflag = 0u, praw = 0u;
            while (true) {
                const unsigned st = cur >> 4, hf = (cur >> 3) & 1u, g2 = cur & 7u;
                const unsigned off = off0 + (hf * NKS + st) * (unsigned)A_BYTES + g2 * 4096u;
                push_1k(off, d[0], wsd);
                const unsigned raw = claim_issue(q, 1u);
                if (pflag != 0u) {
                    wait_vm<2>();
                    if ((claim_get<false>(praw) & 15u) == 15u) push_flag(pflag, call, wsd);
                }
                const unsigned pos = claim_get<true>(raw) + qoff;
                const bool more = pos < Q_ITEMS;
                const unsigned nxt = more ? remap(pos) : cur;
                v4i e[2];
                load_item(nxt, e);
                __builtin_amdgcn_s_sleep(PACE);
                push_1k(off + 1024u, d[1], wsd);
                wait_vm<0>();
                {
                    praw = claim_issue(cnt0 + (hf * NKS + st) * 16u, 1u);
                    pflag = sfoff0 + (hf * NKS + st) * 4u;
                }
                if (!more) break;
                __builtin_amdgcn_s_sleep(PACE);
                cur = nxt;
                d[0] = e[0]; d[1] = e[1];
            }
            if (pflag != 0u && (claim_get<true>(praw) & 15u) == 15u) push_flag(pflag, call, wsd);
        } else {
            __builtin_amdgcn_s_barrier();
        }
        wave_exit();
        return;
    }
    __builtin_amdgcn_s_barrier();
    asm volatile("" ::: "memory");
    const unsigned ctr_v = __hip_atomic_load(ctr, __ATOMIC_RELAXED, __HIP_MEMORY_SCOPE_AGENT);
    int scale_i = __builtin_amdgcn_readfirstlane(__float_as_int(p.sx[0] * p.sw[0]));
    asm volatile("" : "+s"(scale_i));
    const unsigned call = (unsigned)__builtin_amdgcn_readfirstlane(ctr_v) / WGS + 1u;
    const unsigned slot = call & 1u;
#define STAMP(i)
    STAMP(0);

    const int mt = 4 * (xcd & 3) + (j & 3);
    const int nt = 8 * (xcd >> 2) + (j >> 2);
    unsigned* tmo = (unsigned*)(p.ws + OFF_TMO);

    const bool own = (mt >> 1) == p.rank;
    if (wave >= 8) {
        if (own) {
            const unsigned v = NKS;
            asm volatile("ds_write_b32 %0, %1" :: "v"(a_avail), "v"(v) : "memory");
        } else {
            const unsigned* f = (const unsigned*)(p.ws + OFF_FLAGS) + (size_t)mt * NKS + (lane & 31);
            const unsigned long long t0 = __builtin_amdgcn_s_memrealtime();
            unsigned av = 0u;
            for (unsigned it = 1; av < (unsigned)NKS; ++it) {
                const unsigned fv = __hip_atomic_load(f, __ATOMIC_RELAXED, __HIP_MEMORY_SCOPE_SYSTEM);
                const unsigned m = (unsigned)__builtin_amdgcn_ballot_w64(fv < call);
                unsigned v = m == 0u ? (unsigned)NKS : (unsigned)__builtin_ctz(m);
                if ((it & 15u) == 0u && v < (unsigned)NKS &&
                    (__hip_atomic_load(tmo, __ATOMIC_RELAXED, __HIP_MEMORY_SCOPE_AGENT) == call ||
                     __builtin_amdgcn_s_memrealtime() - t0 > DEADLINE_TICKS)) {
                    __hip_atomic_store(tmo, call, __ATOMIC_RELAXED, __HIP_MEMORY_SCOPE_AGENT);
                    asm volatile("ds_write_b32 %0, %1 offset:32" :: "v"(a_avail), "v"(call) : "memory");
                    v = NKS;
                }
                if (v > av) {
                    av = v;
                    asm volatile("ds_write_b32 %0, %1" :: "v"(a_avail), "v"(v) : "memory");
                    if (v < (unsigned)NKS) __builtin_amdgcn_s_sleep(POLL_GAP);
                } else {
                    __builtin_amdgcn_s_sleep(POLL_SLEEP);
                }
            }
        }
        wave_exit();
        return;
    }

    const unsigned char* a_src = p.ws + OFF_STAGE + slot * SLOT_BYTES + (size_t)mt * NKS * A_BYTES;
    const unsigned char* b_src = p.w + (size_t)p.rank * NS + nt * BN;

    const unsigned char* g_ptr;
    int g_odd, g_pair, g_step, l_off;
    if (wave < 4) {
        l_off = wave * 8192;
        if (own) {
            const int row = (mt & 1) * 256 + wave * 64 + (lane >> 3);
            const int ce = (lane & 7) ^ (lane >> 4), co = ce ^ 4;
            g_ptr = p.x + (size_t)row * KDIM + ce * 16;
            g_odd = 8 * KDIM + (co - ce) * 16; g_pair = 16 * KDIM; g_step = BK;
        } else {
            g_ptr = a_src + wave * 8192 + lane * 16;
            g_odd = 1024; g_pair = 2048; g_step = A_BYTES;
        }
    } else {
        const int wb = (wave - 4) & 1;
        const int kr = wb * 64 + (lane >> 3);
        l_off = A_BYTES + wb * 8192;
        const int ch = (lane & 7) ^ (((kr >> 1) & 3) << 1);
        g_ptr = b_src + (size_t)kr * NFULL + ch * 16;
        g_odd = 8 * NFULL; g_pair = 16 * NFULL; g_step = BK * NFULL;
    }
    auto issue_stage = [&](int s, int buf) {
        const unsigned char* g = g_ptr + (size_t)s * g_step;
        unsigned char* l = lds + buf * ST_BYTES + l_off;
        if (wave < 4) {
#pragma unroll
            for (int i = 0; i < 8; ++i)
                __builtin_amdgcn_global_load_lds((const unsigned*)(g + (i >> 1) * g_pair + (i & 1) * g_odd),
                                                 (unsigned*)(l + i * 1024), 16, 0, A_AUX);
        } else if (wave < 6) {
#pragma unroll
            for (int i = 0; i < 8; ++i)
                __builtin_amdgcn_global_load_lds((const unsigned*)(g + (i >> 1) * g_pair + (i & 1) * g_odd),
                                                 (unsigned*)(l + i * 1024), 16, 0, B_AUX);
        }
    };
    auto wait_avail = [&](int s) {
        const unsigned need = (unsigned)s + 1u;
        while (lds_ld(a_avail) < need) __builtin_amdgcn_s_sleep(AV_SLEEP);
    };
    auto wait_older = [&]() {
        wait_vm<8>();
    };
    auto arrive = [&]() {
        if (lane == 0) {
            unsigned one = 1u;
            asm volatile("ds_add_u32 %0, %1" :: "v"(a_cnt), "v"(one) : "memory");
        }
    };

    const int wm = wave >> 1, wn = wave & 1;
    const int h = lane >> 5, r31 = lane & 31;
    const int ax = (r31 >> 1) & 7;
    const int a_base = (wm * 64 + r31) * 128;
    int a_off[2][2];
#pragma unroll
    for (int s = 0; s < 2; ++s)
#pragma unroll
        for (int e = 0; e < 2; ++e) a_off[s][e] = a_base + (((4 * s + 2 * h + e) ^ ax) << 4);
    const int g = lane >> 4, q = (lane & 15) >> 1, pp = lane & 1;
    int b_off[2];
#pragma unroll
    for (int nb = 0; nb < 2; ++nb)
        b_off[nb] = A_BYTES + (32 * h + q) * 128 + (((wn * 4 + nb * 2 + (g & 1)) ^ (((q >> 1) & 3) << 1)) << 4) + pp * 8;

    v16f acc[2][2];
#pragma unroll
    for (int i = 0; i < 2; ++i)
#pragma unroll
        for (int k = 0; k < 2; ++k)
#pragma unroll
            for (int r = 0; r < 16; ++r) acc[i][k][r] = 0.f;

    if (wave < 4) wait_avail(0);
    STAMP(1);
    issue_stage(0, 0);
    if (wave < 4) wait_avail(1);
    issue_stage(1, 1);
    wait_older();
    arrive();

    int buf = 0;
    for (int s = 0; s < NKS; ++s) {
        const int s2 = s + 2;
        int nbuf = buf + 2; if (nbuf >= NBUF) nbuf -= NBUF;
        {
            const unsigned target = 8u * (unsigned)(s + 1);
            while (lds_ld(a_cnt) < target) {}
        }
        bool issued = true;
        if (s2 < NKS) {
            if (wave < 4) issued = lds_ld(a_avail) >= (unsigned)s2 + 1u;
            if (issued) issue_stage(s2, nbuf);
        }

        const unsigned lb = lds_base + buf * ST_BYTES;
        v4i alo[2][2], ahi[2][2];
        v2i bt[2][2][4];
        {
            const unsigned a0 = lb + a_off[0][0], a1 = lb + a_off[0][1];
            const unsigned a2 = lb + a_off[1][0], a3 = lb + a_off[1][1];
            const unsigned b0 = lb + b_off[0], b1 = lb + b_off[1];
            DS_B128(alo[0][0], a0, 0);    DS_B128(ahi[0][0], a1, 0);
            DS_TR8(bt[0][0][0], b0, 0);   DS_TR8(bt[0][0][1], b0, 1024);
            DS_TR8(bt[0][0][2], b0, 2048); DS_TR8(bt[0][0][3], b0, 3072);
            DS_B128(alo[0][1], a0, 4096); DS_B128(ahi[0][1], a1, 4096);
            DS_TR8(bt[0][1][0], b1, 0);   DS_TR8(bt[0][1][1], b1, 1024);
            DS_TR8(bt[0][1][2], b1, 2048); DS_TR8(bt[0][1][3], b1, 3072);
            DS_B128(alo[1][0], a2, 0);    DS_B128(ahi[1][0], a3, 0);
            DS_TR8(bt[1][0][0], b0, 8192); DS_TR8(bt[1][0][1], b0, 9216);
            DS_TR8(bt[1][0][2], b0, 10240); DS_TR8(bt[1][0][3], b0, 11264);
            DS_B128(alo[1][1], a2, 4096); DS_B128(ahi[1][1], a3, 4096);
            DS_TR8(bt[1][1][0], b1, 8192); DS_TR8(bt[1][1][1], b1, 9216);
            DS_TR8(bt[1][1][2], b1, 10240); DS_TR8(bt[1][1][3], b1, 11264);
        }
#pragma unroll
        for (int ks = 0; ks < 2; ++ks) {
            if (ks == 0) {
                asm volatile("s_waitcnt lgkmcnt(12)" ::: "memory");
            } else {
                asm volatile("s_waitcnt lgkmcnt(0)" ::: "memory");
                if (s + 1 < NKS) {
                    if (!issued && s >= VMFROM) {
                        while ((__builtin_amdgcn_s_getreg(7 | (31 << 11)) & 0x00C0000Fu) != 0u) {
                            if (lds_ld(a_avail) >= (unsigned)s2 + 1u) { issue_stage(s2, nbuf); issued = true; break; }
                        }
                    }
                    if (s2 < NKS && issued) wait_older(); else wait_vm<0>();
                    arrive();
                }
            }
            __builtin_amdgcn_sched_barrier(0);
            v8i af[2], bf[2];
#pragma unroll
            for (int mb = 0; mb < 2; ++mb)
                af[mb] = __builtin_shufflevector(alo[ks][mb], ahi[ks][mb], 0, 1, 2, 3, 4, 5, 6, 7);
#pragma unroll
            for (int nb = 0; nb < 2; ++nb) {
                v4i lo = __builtin_shufflevector(bt[ks][nb][0], bt[ks][nb][1], 0, 1, 2, 3);
                v4i hi = __builtin_shufflevector(bt[ks][nb][2], bt[ks][nb][3], 0, 1, 2, 3);
                bf[nb] = __builtin_shufflevector(lo, hi, 0, 1, 2, 3, 4, 5, 6, 7);
            }
#pragma unroll
            for (int mb = 0; mb < 2; ++mb)
#pragma unroll
                for (int nb = 0; nb < 2; ++nb)
                    acc[mb][nb] = __builtin_amdgcn_mfma_scale_f32_32x32x64_f8f6f4(
                        af[mb], bf[nb], acc[mb][nb], 0, 0, 0, 0x7f7f7f7f, 0, 0x7f7f7f7f);
            __builtin_amdgcn_sched_barrier(0);
        }
        if (!issued) { wait_avail(s2); issue_stage(s2, nbuf); }
        buf = buf + 1; if (buf >= NBUF) buf = 0;
    }

    STAMP(2);
    const bool bad = lds_ld(a_avail + 32) != 0u;
    const float scale = __int_as_float(scale_i);
    const float nanv = __builtin_nanf("");
    float* o = p.out + (size_t)(mt * BM + wm * 64 + 4 * h) * NS + nt * BN + wn * 64 + r31;
#pragma unroll
    for (int mb = 0; mb < 2; ++mb)
#pragma unroll
        for (int nb = 0; nb < 2; ++nb)
#pragma unroll
            for (int r = 0; r < 16; ++r) {
                const int row = mb * 32 + (r & 3) + 8 * (r >> 2);
                float v = fmaxf(acc[mb][nb][r] * scale, 0.f);
                __hip_atomic_store(&o[(size_t)row * NS + nb * 32], bad ? nanv : v, __ATOMIC_RELAXED, __HIP_MEMORY_SCOPE_AGENT);
            }
    STAMP(3);
    wave_exit();
}

}

extern "C" void kernel_launch(int rank, int nranks, ncclComm_t comm,
                              hipStream_t stream,
                              void* const* d_in, const int* in_sizes, int n_in,
                              void* d_out, int out_size,
                              const int* in_dtypes, int out_dtype,
                              void* const* d_out_all,
                              void* const* d_ws_all, size_t ws_bytes) {
    (void)comm; (void)in_sizes; (void)n_in; (void)out_size; (void)in_dtypes; (void)out_dtype; (void)d_out_all;
    if (nranks != NR || ws_bytes < WS_NEED || (unsigned)rank >= (unsigned)NR) {
        fprintf(stderr, "unexpected configuration\n");
        return;
    }
    HIP_CHECK(hipFuncSetAttribute((const void*)fused_kernel, hipFuncAttributeMaxDynamicSharedMemorySize, LDS_BYTES));
    Params p;
    p.x = (const unsigned char*)d_in[0];
    p.w = (const unsigned char*)d_in[1];
    p.sx = (const float*)d_in[2];
    p.sw = (const float*)d_in[3];
    p.out = (float*)d_out;
    p.ws = (unsigned char*)d_ws_all[rank];
    for (int i = 0; i < NR; ++i) p.peer_ws[i] = (unsigned char*)d_ws_all[(unsigned)(rank + 1 + i) & 7u];
    p.rank = rank;
    fused_kernel<<<WGS, THREADS, LDS_BYTES, stream>>>(p);
}
```

```cpp
#include <hip/hip_runtime.h>
#include <rccl/rccl.h>
#include <cstdio>
#include <cstdint>

#define HIP_CHECK(expr) do { hipError_t _e = (expr); if (_e != hipSuccess) { \
    fprintf(stderr, "HIP error %s at %s:%d\n", hipGetErrorString(_e), __FILE__, __LINE__); \
    return; } } while (0)

namespace {

typedef int   v8i  __attribute__((ext_vector_type(8)));
typedef int   v4i  __attribute__((ext_vector_type(4)));
typedef int   v2i  __attribute__((ext_vector_type(2)));
typedef float v16f __attribute__((ext_vector_type(16)));

constexpr int NR    = 8;
constexpr int KDIM  = 4096;
constexpr int NFULL = 16384;
constexpr int NS    = 2048;
constexpr int BM = 256, BN = 128, BK = 128;
constexpr int NKS   = KDIM / BK;
#define CHST 2
constexpr int CH_ST = CHST;
constexpr int NCH   = NKS / CH_ST;
constexpr unsigned Q_ITEMS = NCH * 32;
constexpr unsigned Q_WAVES = 32;
#define PACE 6
#define POLL_GAP 20
#define POLL_SLEEP 6
#define LEAD 2u
#define VMFROM 26
#define INIT_TID 128
#define PRIO_S 3
#define AV_SLEEP 1
#define A_AUX 17
#define B_AUX 0
constexpr int A_BYTES = BM * BK;
constexpr int B_BYTES = BK * BN;
constexpr int ST_BYTES = A_BYTES + B_BYTES;
constexpr int NBUF = 3;
constexpr int SYNC_OFF = NBUF * ST_BYTES;
constexpr int LDS_BYTES = SYNC_OFF + 64;

constexpr int WGS = 256, THREADS = 704;

constexpr size_t OFF_CTR   = 0;
constexpr size_t OFF_TMO   = 512;
constexpr size_t OFF_QUEUE = 4096;
constexpr size_t OFF_FLAGS = 8192;
constexpr size_t OFF_SCNT  = 65536;
constexpr size_t OFF_STAGE = 1u << 20;
constexpr size_t SLOT_BYTES = (size_t)NR * 2 * NKS * A_BYTES;
constexpr size_t WS_NEED   = OFF_STAGE + 2 * SLOT_BYTES;

constexpr unsigned long long DEADLINE_TICKS = 150000000ull;

template <int N> __device__ __forceinline__ void wait_vm() {
    asm volatile("s_waitcnt vmcnt(%0)" :: "n"(N) : "memory");
}
#define DS_B128(dst, addr, off) asm volatile("ds_read_b128 %0, %1 offset:" #off : "=v"(dst) : "v"(addr) : "memory")
#define DS_TR8(dst, addr, off)  asm volatile("ds_read_b64_tr_b8 %0, %1 offset:" #off : "=v"(dst) : "v"(addr) : "memory")

__device__ __forceinline__ unsigned lds_ld(unsigned addr) {
    unsigned v;
    asm volatile("ds_read_b32 %0, %1\n\ts_waitcnt lgkmcnt(0)" : "=v"(v) : "v"(addr) : "memory");
    return (unsigned)__builtin_amdgcn_readfirstlane(v);
}

struct Params {
    const unsigned char* x;
    const unsigned char* w;
    const float* sx;
    const float* sw;
    float* out;
    unsigned char* ws;
    unsigned char* peer_ws[NR];
    int rank;
};

__device__ __forceinline__ unsigned claim_issue(unsigned* q, unsigned inc) {
    unsigned r;
    const unsigned zero = 0u;
    asm volatile("s_mov_b64 exec, 1\n\tglobal_atomic_add %0, %1, %2, %3 sc0\n\ts_mov_b64 exec, -1"
                 : "=&v"(r) : "v"(zero), "v"(inc), "s"(q) : "memory");
    return r;
}
template <bool WAIT> __device__ __forceinline__ unsigned claim_get(unsigned r) {
    unsigned v;
    if constexpr (WAIT) asm volatile("s_waitcnt vmcnt(0)\n\tv_readfirstlane_b32 %0, %1\n\ts_nop 4" : "=s"(v) : "v"(r) : "memory");
    else                asm volatile("v_readfirstlane_b32 %0, %1\n\ts_nop 4" : "=s"(v) : "v"(r) : "memory");
    return v;
}
__device__ __forceinline__ void push_1k(unsigned off, v4i d, const unsigned char* base) {
    asm volatile("global_store_dwordx4 %0, %1, %2 sc0 sc1\n\ts_nop 1" :: "v"(off), "v"(d), "s"(base) : "memory");
}
__device__ __forceinline__ void push_flag(unsigned off, unsigned v, const unsigned char* base) {
    asm volatile("global_store_dword %0, %1, %2 sc0 sc1\n\ts_nop 1" :: "v"(off), "v"(v), "s"(base) : "memory");
}

__global__ void __launch_bounds__(THREADS, 3) fused_kernel(Params pa) {
    extern __shared__ __attribute__((aligned(16))) unsigned char lds[];
    const int tid  = threadIdx.x;
    const int lane = tid & 63;
    const int wave_hw = __builtin_amdgcn_readfirstlane(tid >> 6);
    const int wave = wave_hw < 3 ? wave_hw + 8 : wave_hw - 3;
    const int b = blockIdx.x;
    struct { const unsigned char* x; const unsigned char* w; const float* sx; const float* sw; float* out;
             unsigned char* ws; unsigned char* wsd; int rank; } p;
    {
        typedef __attribute__((address_space(1))) unsigned char gu8;
        gu8* gx = (gu8*)pa.x; gu8* gw = (gu8*)pa.w; gu8* gsx = (gu8*)pa.sx; gu8* gsw = (gu8*)pa.sw;
        gu8* go = (gu8*)pa.out; gu8* gws = (gu8*)pa.ws; gu8* gwsd = (gu8*)pa.peer_ws[(unsigned)((b >> 3) + b) & 7u];
        int rk = pa.rank;
        asm volatile("" : "+s"(gx), "+s"(gw), "+s"(gsx), "+s"(gsw), "+s"(go), "+s"(gws), "+s"(gwsd), "+s"(rk));
        p.x = (const unsigned char*)gx; p.w = (const unsigned char*)gw; p.sx = (const float*)gsx; p.sw = (const float*)gsw;
        p.out = (float*)go; p.ws = (unsigned char*)gws; p.wsd = (unsigned char*)gwsd; p.rank = rk;
    }
    const unsigned lds_base = (unsigned)(size_t)(__attribute__((address_space(3))) unsigned char*)lds;
    const unsigned a_cnt = lds_base + SYNC_OFF, a_avail = lds_base + SYNC_OFF + 16;

    const int xcd = b & 7, j = b >> 3;
    unsigned* ctr = (unsigned*)(p.ws + OFF_CTR);
    const unsigned a_exit = lds_base + SYNC_OFF + 32;
    if (tid == INIT_TID) {
        const unsigned z = 0u;
        asm volatile("ds_write_b32 %0, %1\n\tds_write_b32 %0, %1 offset:16\n\tds_write_b32 %0, %1 offset:32\n\t"
                     "ds_write_b32 %0, %1 offset:48\n\tds_write_b32 %0, %1 offset:4\n\ts_waitcnt lgkmcnt(0)" :: "v"(a_cnt), "v"(z) : "memory");
    }
    auto wave_exit = [&]() {
        if (lane == 0) {
            unsigned old;
            const unsigned one = 1u;
            asm volatile("ds_add_rtn_u32 %0, %1, %2\n\ts_waitcnt lgkmcnt(0)" : "=v"(old) : "v"(a_exit), "v"(one) : "memory");
            if (old == (unsigned)(THREADS / 64 - 1))
                __hip_atomic_fetch_add(ctr, 1u, __ATOMIC_RELAXED, __HIP_MEMORY_SCOPE_AGENT);
        }
    };
    if (wave >= 8) __builtin_amdgcn_s_setprio(PRIO_S);
    if (wave == 8 || wave == 9) {
        unsigned ctr_v = __hip_atomic_load(ctr, __ATOMIC_RELAXED, __HIP_MEMORY_SCOPE_AGENT);
        const int di = (j + xcd) & 7;
        if (di < NR - 1) {
            const int sq = wave & 1;
            unsigned* q = (unsigned*)(p.ws + OFF_QUEUE + (size_t)(di * 2 + sq) * 256);
            const unsigned char* wsd = p.wsd;
            const unsigned lsrc = (unsigned)((sq * 16 + (lane >> 3)) * KDIM + (((lane & 7) ^ (lane >> 4)) << 4));
            auto remap = [&](unsigned pos) -> unsigned {
                const unsigned gi = pos >> 3, t = gi - LEAD;
                unsigned st, hf;
                if (gi < LEAD) { hf = 0u; st = gi; }
                else if (gi < 64u - LEAD) { hf = (t & 1u) ^ 1u; st = (t >> 1) + ((t & 1u) ? LEAD : 0u); }
                else { hf = 1u; st = gi - 32u; }
                return (st << 4) | (hf << 3) | (pos & 7u);
            };
            auto load_item = [&](unsigned it, v4i* dst) {
                const unsigned char* sp = p.x + (size_t)((it & 15u) * (32u * KDIM) + (it >> 4) * BK);
                dst[0] = *(const v4i*)(sp + lsrc);
                dst[1] = *(const v4i*)(sp + 8 * KDIM + (lsrc ^ 64u));
            };
            unsigned cur = remap((unsigned)(xcd * 4 + (j >> 3)));
            v4i d[2];
            load_item(cur, d);
            __builtin_amdgcn_s_barrier();
            asm volatile("" : "+v"(ctr_v));
            const unsigned call = (unsigned)__builtin_amdgcn_readfirstlane(ctr_v) / WGS + 1u;
            const unsigned qoff = Q_WAVES - (call - 1u) * Q_ITEMS;
            const unsigned off0 = (unsigned)(OFF_STAGE + (size_t)p.rank * 2 * NKS * A_BYTES) + (call & 1u) * (unsigned)SLOT_BYTES +
                                  (unsigned)(sq * 2048 + lane * 16);
            asm volatile("" :: "v"(d[0]), "v"(d[1]));
            unsigned* cnt0 = (unsigned*)(p.ws + OFF_SCNT + (size_t)di * 64 * 64);
            const unsigned sfoff0 = (unsigned)OFF_FLAGS + (unsigned)(p.rank * 2 * NKS) * 4u;
            unsigned pflag = 0u, praw = 0u;
            while (true) {
                const unsigned st = cur >> 4, hf = (cur >> 3) & 1u, g2 = cur & 7u;
                const unsigned off = off0 + (hf * NKS + st) * (unsigned)A_BYTES + g2 * 4096u;
                push_1k(off, d[0], wsd);
                const unsigned raw = claim_issue(q, 1u);
                if (pflag != 0u) {
                    wait_vm<2>();
                    if ((claim_get<false>(praw) & 15u) == 15u) push_flag(pflag, call, wsd);
                }
                const unsigned pos = claim_get<true>(raw) + qoff;
                const bool more = pos < Q_ITEMS;
                const unsigned nxt = more ? remap(pos) : cur;
                v4i e[2];
                load_item(nxt, e);
                __builtin_amdgcn_s_sleep(PACE);
                push_1k(off + 1024u, d[1], wsd);
                wait_vm<0>();
                praw = claim_issue(cnt0 + (hf * NKS + st) * 16u, 1u);
                pflag = sfoff0 + (hf * NKS + st) * 4u;
                if (!more) break;
                __builtin_amdgcn_s_sleep(PACE);
                cur = nxt;
                d[0] = e[0]; d[1] = e[1];
            }
            if (pflag != 0u && (claim_get<true>(praw) & 15u) == 15u) push_flag(pflag, call, wsd);
        } else {
            __builtin_amdgcn_s_barrier();
        }
        wave_exit();
        return;
    }
    __builtin_amdgcn_s_barrier();
    asm volatile("" ::: "memory");
    const unsigned ctr_v = __hip_atomic_load(ctr, __ATOMIC_RELAXED, __HIP_MEMORY_SCOPE_AGENT);
    int scale_i = __builtin_amdgcn_readfirstlane(__float_as_int(p.sx[0] * p.sw[0]));
    asm volatile("" : "+s"(scale_i));
    const unsigned call = (unsigned)__builtin_amdgcn_readfirstlane(ctr_v) / WGS + 1u;
    const unsigned slot = call & 1u;

    const int mt = 4 * (xcd & 3) + (j & 3);
    const int nt = 8 * (xcd >> 2) + (j >> 2);
    unsigned* tmo = (unsigned*)(p.ws + OFF_TMO);

    const bool own = (mt >> 1) == p.rank;
    if (wave >= 8) {
        if (own) {
            const unsigned v = NKS;
            asm volatile("ds_write_b32 %0, %1" :: "v"(a_avail), "v"(v) : "memory");
        } else {
            const unsigned* f = (const unsigned*)(p.ws + OFF_FLAGS) + (size_t)mt * NKS + (lane & 31);
            const unsigned long long t0 = __builtin_amdgcn_s_memrealtime();
            unsigned av = 0u;
            for (unsigned it = 1; av < (unsigned)NKS; ++it) {
                const unsigned fv = __hip_atomic_load(f, __ATOMIC_RELAXED, __HIP_MEMORY_SCOPE_SYSTEM);
                const unsigned m = (unsigned)__builtin_amdgcn_ballot_w64(fv < call);
                unsigned v = m == 0u ? (unsigned)NKS : (unsigned)__builtin_ctz(m);
                if ((it & 15u) == 0u && v < (unsigned)NKS &&
                    (__hip_atomic_load(tmo, __ATOMIC_RELAXED, __HIP_MEMORY_SCOPE_AGENT) == call ||
                     __builtin_amdgcn_s_memrealtime() - t0 > DEADLINE_TICKS)) {
                    __hip_atomic_store(tmo, call, __ATOMIC_RELAXED, __HIP_MEMORY_SCOPE_AGENT);
                    asm volatile("ds_write_b32 %0, %1 offset:32" :: "v"(a_avail), "v"(call) : "memory");
                    v = NKS;
                }
                if (v > av) {
                    av = v;
                    asm volatile("ds_write_b32 %0, %1" :: "v"(a_avail), "v"(v) : "memory");
                    if (v < (unsigned)NKS) __builtin_amdgcn_s_sleep(POLL_GAP);
                } else {
                    __builtin_amdgcn_s_sleep(POLL_SLEEP);
                }
            }
        }
        wave_exit();
        return;
    }

    const unsigned char* a_src = p.ws + OFF_STAGE + slot * SLOT_BYTES + (size_t)mt * NKS * A_BYTES;
    const unsigned char* b_src = p.w + (size_t)p.rank * NS + nt * BN;

    const unsigned char* g_ptr;
    int g_odd, g_pair, g_step, l_off;
    if (wave < 4) {
        l_off = wave * 8192;
        if (own) {
            const int row = (mt & 1) * 256 + wave * 64 + (lane >> 3);
            const int ce = (lane & 7) ^ (lane >> 4), co = ce ^ 4;
            g_ptr = p.x + (size_t)row * KDIM + ce * 16;
            g_odd = 8 * KDIM + (co - ce) * 16; g_pair = 16 * KDIM; g_step = BK;
        } else {
            g_ptr = a_src + wave * 8192 + lane * 16;
            g_odd = 1024; g_pair = 2048; g_step = A_BYTES;
        }
    } else {
        const int wb = (wave - 4) & 1;
        const int kr = wb * 64 + (lane >> 3);
        l_off = A_BYTES + wb * 8192;
        const int ch = (lane & 7) ^ (((kr >> 1) & 3) << 1);
        g_ptr = b_src + (size_t)kr * NFULL + ch * 16;
        g_odd = 8 * NFULL; g_pair = 16 * NFULL; g_step = BK * NFULL;
    }
    auto issue_stage = [&](int s, int buf) {
        const unsigned char* g = g_ptr + (size_t)s * g_step;
        unsigned char* l = lds + buf * ST_BYTES + l_off;
        if (wave < 4) {
#pragma unroll
            for (int i = 0; i < 8; ++i)
                __builtin_amdgcn_global_load_lds((const unsigned*)(g + (i >> 1) * g_pair + (i & 1) * g_odd),
                                                 (unsigned*)(l + i * 1024), 16, 0, A_AUX);
        } else if (wave < 6) {
#pragma unroll
            for (int i = 0; i < 8; ++i)
                __builtin_amdgcn_global_load_lds((const unsigned*)(g + (i >> 1) * g_pair + (i & 1) * g_odd),
                                                 (unsigned*)(l + i * 1024), 16, 0, B_AUX);
        }
    };
    auto wait_avail = [&](int s) {
        const unsigned need = (unsigned)s + 1u;
        while (lds_ld(a_avail) < need) __builtin_amdgcn_s_sleep(AV_SLEEP);
    };
    auto wait_older = [&]() {
        wait_vm<8>();
    };
    auto arrive = [&]() {
        if (lane == 0) {
            unsigned one = 1u;
            asm volatile("ds_add_u32 %0, %1" :: "v"(a_cnt), "v"(one) : "memory");
        }
    };

    const int wm = wave >> 1, wn = wave & 1;
    const int h = lane >> 5, r31 = lane & 31;
    const int ax = (r31 >> 1) & 7;
    const int a_base = (wm * 64 + r31) * 128;
    int a_off[2][2];
#pragma unroll
    for (int s = 0; s < 2; ++s)
#pragma unroll
        for (int e = 0; e < 2; ++e) a_off[s][e] = a_base + (((4 * s + 2 * h + e) ^ ax) << 4);
    const int g = lane >> 4, q = (lane & 15) >> 1, pp = lane & 1;
    int b_off[2];
#pragma unroll
    for (int nb = 0; nb < 2; ++nb)
        b_off[nb] = A_BYTES + (32 * h + q) * 128 + (((wn * 4 + nb * 2 + (g & 1)) ^ (((q >> 1) & 3) << 1)) << 4) + pp * 8;

    v16f acc[2][2];
#pragma unroll
    for (int i = 0; i < 2; ++i)
#pragma unroll
        for (int k = 0; k < 2; ++k)
#pragma unroll
            for (int r = 0; r < 16; ++r) acc[i][k][r] = 0.f;

    if (wave < 4) wait_avail(0);
    issue_stage(0, 0);
    if (wave < 4) wait_avail(1);
    issue_stage(1, 1);
    wait_older();
    arrive();

    int buf = 0;
    for (int s = 0; s < NKS; ++s) {
        const int s2 = s + 2;
        int nbuf = buf + 2; if (nbuf >= NBUF) nbuf -= NBUF;
        {
            const unsigned target = 8u * (unsigned)(s + 1);
            while (lds_ld(a_cnt) < target) {}
        }
        bool issued = true;
        if (s2 < NKS) {
            if (wave < 4) issued = lds_ld(a_avail) >= (unsigned)s2 + 1u;
            if (issued) issue_stage(s2, nbuf);
        }

        const unsigned lb = lds_base + buf * ST_BYTES;
        v4i alo[2][2], ahi[2][2];
        v2i bt[2][2][4];
        {
            const unsigned a0 = lb + a_off[0][0], a1 = lb + a_off[0][1];
            const unsigned a2 = lb + a_off[1][0], a3 = lb + a_off[1][1];
            const unsigned b0 = lb + b_off[0], b1 = lb + b_off[1];
            DS_B128(alo[0][0], a0, 0);    DS_B128(ahi[0][0], a1, 0);
            DS_TR8(bt[0][0][0], b0, 0);   DS_TR8(bt[0][0][1], b0, 1024);
            DS_TR8(bt[0][0][2], b0, 2048); DS_TR8(bt[0][0][3], b0, 3072);
            DS_B128(alo[0][1], a0, 4096); DS_B128(ahi[0][1], a1, 4096);
            DS_TR8(bt[0][1][0], b1, 0);   DS_TR8(bt[0][1][1], b1, 1024);
            DS_TR8(bt[0][1][2], b1, 2048); DS_TR8(bt[0][1][3], b1, 3072);
            DS_B128(alo[1][0], a2, 0);    DS_B128(ahi[1][0], a3, 0);
            DS_TR8(bt[1][0][0], b0, 8192); DS_TR8(bt[1][0][1], b0, 9216);
            DS_TR8(bt[1][0][2], b0, 10240); DS_TR8(bt[1][0][3], b0, 11264);
            DS_B128(alo[1][1], a2, 4096); DS_B128(ahi[1][1], a3, 4096);
            DS_TR8(bt[1][1][0], b1, 8192); DS_TR8(bt[1][1][1], b1, 9216);
            DS_TR8(bt[1][1][2], b1, 10240); DS_TR8(bt[1][1][3], b1, 11264);
        }
#pragma unroll
        for (int ks = 0; ks < 2; ++ks) {
            if (ks == 0) {
                asm volatile("s_waitcnt lgkmcnt(12)" ::: "memory");
            } else {
                asm volatile("s_waitcnt lgkmcnt(0)" ::: "memory");
                if (s + 1 < NKS) {
                    if (!issued && s >= VMFROM) {
                        while ((__builtin_amdgcn_s_getreg(7 | (31 << 11)) & 0x00C0000Fu) != 0u) {
                            if (lds_ld(a_avail) >= (unsigned)s2 + 1u) { issue_stage(s2, nbuf); issued = true; break; }
                        }
                    }
                    if (s2 < NKS && issued) wait_older(); else wait_vm<0>();
                    arrive();
                }
            }
            __builtin_amdgcn_sched_barrier(0);
            v8i af[2], bf[2];
#pragma unroll
            for (int mb = 0; mb < 2; ++mb)
                af[mb] = __builtin_shufflevector(alo[ks][mb], ahi[ks][mb], 0, 1, 2, 3, 4, 5, 6, 7);
#pragma unroll
            for (int nb = 0; nb < 2; ++nb) {
                v4i lo = __builtin_shufflevector(bt[ks][nb][0], bt[ks][nb][1], 0, 1, 2, 3);
                v4i hi = __builtin_shufflevector(bt[ks][nb][2], bt[ks][nb][3], 0, 1, 2, 3);
                bf[nb] = __builtin_shufflevector(lo, hi, 0, 1, 2, 3, 4, 5, 6, 7);
            }
#pragma unroll
            for (int mb = 0; mb < 2; ++mb)
#pragma unroll
                for (int nb = 0; nb < 2; ++nb)
                    acc[mb][nb] = __builtin_amdgcn_mfma_scale_f32_32x32x64_f8f6f4(
                        af[mb], bf[nb], acc[mb][nb], 0, 0, 0, 0x7f7f7f7f, 0, 0x7f7f7f7f);
            __builtin_amdgcn_sched_barrier(0);
        }
        if (!issued) { wait_avail(s2); issue_stage(s2, nbuf); }
        buf = buf + 1; if (buf >= NBUF) buf = 0;
    }
    const bool bad = lds_ld(a_avail + 32) != 0u;
    const float scale = __int_as_float(scale_i);
    const float nanv = __builtin_nanf("");
    float* o = p.out + (size_t)(mt * BM + wm * 64 + 4 * h) * NS + nt * BN + wn * 64 + r31;
#pragma unroll
    for (int mb = 0; mb < 2; ++mb)
#pragma unroll
        for (int nb = 0; nb < 2; ++nb)
#pragma unroll
            for (int r = 0; r < 16; ++r) {
                const int row = mb * 32 + (r & 3) + 8 * (r >> 2);
                float v = fmaxf(acc[mb][nb][r] * scale, 0.f);
                __hip_atomic_store(&o[(size_t)row * NS + nb * 32], bad ? nanv : v, __ATOMIC_RELAXED, __HIP_MEMORY_SCOPE_AGENT);
            }
    wave_exit();
}

}

extern "C" void kernel_launch(int rank, int nranks, ncclComm_t comm,
                              hipStream_t stream,
                              void* const* d_in, const int* in_sizes, int n_in,
                              void* d_out, int out_size,
                              const int* in_dtypes, int out_dtype,
                              void* const* d_out_all,
                              void* const* d_ws_all, size_t ws_bytes) {
    (void)comm; (void)in_sizes; (void)n_in; (void)out_size; (void)in_dtypes; (void)out_dtype; (void)d_out_all;
    if (nranks != NR || ws_bytes < WS_NEED || (unsigned)rank >= (unsigned)NR) {
        fprintf(stderr, "unexpected configuration\n");
        return;
    }
    HIP_CHECK(hipFuncSetAttribute((const void*)fused_kernel, hipFuncAttributeMaxDynamicSharedMemorySize, LDS_BYTES));
    Params p;
    p.x = (const unsigned char*)d_in[0];
    p.w = (const unsigned char*)d_in[1];
    p.sx = (const float*)d_in[2];
    p.sw = (const float*)d_in[3];
    p.out = (float*)d_out;
    p.ws = (unsigned char*)d_ws_all[rank];
    for (int i = 0; i < NR; ++i) p.peer_ws[i] = (unsigned char*)d_ws_all[(unsigned)(rank + 1 + i) & 7u];
    p.rank = rank;
    fused_kernel<<<WGS, THREADS, LDS_BYTES, stream>>>(p);
}
```
